```python
import numpy as np

kernel_name = "gpu_gemm_a2a_m4096_k4096_n4096_e4m3e5m2_dequant"


ABSMAX_RTOL = 0.05

COMPUTE_DTYPE = "float32"
EPILOGUE = "dequant"
IN_QUANT = None
OUT_QUANT = None
IN_FP8 = "e4m3e5m2"
IN_DTYPES = {"x": "e4m3", "w_mat": "e5m2",
             "scale_x": "float32", "scale_w": "float32"}
WORLD_SIZE = 8

def _snap_fp8(x, mant_bits, max_abs, min_normal):
    a = np.abs(x.astype(np.float64))
    _m, _e = np.frexp(a)
    q = np.where(a >= min_normal, np.ldexp(1.0, _e - 1 - mant_bits),
                 min_normal * 2.0 ** -mant_bits)
    snapped = np.minimum(np.round(a / q) * q, max_abs)
    return (np.sign(x) * snapped).astype(np.float32)

def setup_inputs():

    x = _snap_fp8(np.random.randn(4096, 4096).astype(np.float32), 3, 448.0, 2.0 ** -6)
    w_mat = _snap_fp8(np.random.randn(4096, 4096).astype(np.float32), 2, 57344.0, 2.0 ** -14)
    scale_x = (0.02 * (1.0 + np.random.rand(1))).astype(np.float32)
    scale_w = (0.02 * (1.0 + np.random.rand(1))).astype(np.float32)
    return {"x": x, "w_mat": w_mat, "scale_x": scale_x, "scale_w": scale_w}

def reference(x, w_mat, scale_x, scale_w):
    acc = (x @ w_mat).astype(np.float32)
    y = (acc * (scale_x * scale_w)).astype(np.float32)
    return y
```

```cpp
#include <hip/hip_runtime.h>
#include <rccl/rccl.h>
#include <cstdio>
#include <cstdint>

#define HIP_CHECK(expr) do { hipError_t _e = (expr); if (_e != hipSuccess) { \
    fprintf(stderr, "HIP error %s at %s:%d\n", hipGetErrorString(_e), __FILE__, __LINE__); \
    return; } } while (0)

typedef int i32x8 __attribute__((ext_vector_type(8)));
typedef int i32x4 __attribute__((ext_vector_type(4)));
typedef int i32x2 __attribute__((ext_vector_type(2)));
typedef unsigned u32x4 __attribute__((ext_vector_type(4)));
typedef float f32x16 __attribute__((ext_vector_type(16)));
typedef float f32x4 __attribute__((ext_vector_type(4)));
#define LAS __attribute__((address_space(3)))

constexpr int NR = 8;
constexpr int KDIM = 4096, NDIM = 4096, MLOC = 512, NLOC = 512;
constexpr int BM = 64, BN = 128, BK = 128;
constexpr int NSTEP = KDIM / BK;
constexpr int NS = 6;
constexpr int STAGE = (BM + BN) * BK;
constexpr int LDS_BYTES = NS * STAGE;
constexpr int NWG = 256;

constexpr size_t WS_FLAG = 0;
constexpr size_t WS_CNT = 4096;
constexpr size_t WS_TMO = 8192;
constexpr size_t WS_NEED = 16384;
#define STAMP_AT(k) do {} while (0)
constexpr unsigned long long DEADLINE_TICKS = 50000000ull;

struct Params {
    const uint8_t* x;
    const uint8_t* w;
    const float* sx;
    const float* sw;
    float* out[NR];
    char* ws[NR];
    int rank;
};

__device__ __forceinline__ void st16_sys(void* p, f32x4 v) {
    asm volatile("global_store_dwordx4 %0, %1, off sc0 sc1\n\ts_nop 1" :: "v"(p), "v"(v) : "memory");
}
#define WAIT_V(n) asm volatile("s_waitcnt vmcnt(" #n ")" ::: "memory")
#define WAIT_L0() do { __builtin_amdgcn_s_waitcnt(0xC07F); asm volatile("" ::: "memory"); } while (0)
#define BAR() asm volatile("s_barrier" ::: "memory")
#define SCHED() __builtin_amdgcn_sched_barrier(0)

__device__ __forceinline__ void glds16(const uint8_t* g, unsigned char* l) {
    __builtin_amdgcn_global_load_lds((const unsigned*)g, (LAS unsigned*)l, 16, 0, 0);
}

__global__ void __launch_bounds__(512, 2) gemm_a2a(Params P) {
    extern __shared__ __attribute__((aligned(16))) unsigned char smem[];
    const int lane = threadIdx.x & 63;
    const int wv = __builtin_amdgcn_readfirstlane(threadIdx.x >> 6);
    const int b = blockIdx.x;
    const int xcd = b & 7, ti = b >> 3;
    const int cb = xcd * 4 + (ti & 3);
    const int rb = ti >> 2;
    const int dst = cb >> 2;
    const int rank = P.rank;

    if (wv >= 4) {
        const int lw = wv - 4;
        const int r8 = lane >> 3, c8 = lane & 7;
        const uint8_t* ga0 = P.x + (size_t)(rb * BM + (2 * lw) * 8 + r8) * KDIM + ((c8 ^ ((r8 >> 1) & 7)) << 4);
        const uint8_t* ga1 = P.x + (size_t)(rb * BM + (2 * lw + 1) * 8 + r8) * KDIM + ((c8 ^ ((4 + (r8 >> 1)) & 7)) << 4);
        const uint8_t* gb = P.w + (size_t)(lw * 32 + r8) * NDIM + cb * BN + ((c8 ^ (((r8 >> 1) & 3) << 1)) << 4);
        unsigned char* la = smem + lw * 2048;
        unsigned char* lb = smem + BM * BK + lw * 4096;
        auto issue = [&](int s, int buf) {
            unsigned char* a = la + buf * STAGE;
            unsigned char* bb = lb + buf * STAGE;
            const uint8_t* pb = gb + (size_t)s * BK * NDIM;
            glds16(pb, bb);
            glds16(pb + 8 * NDIM, bb + 1024);
            glds16(pb + 16 * NDIM, bb + 2048);
            glds16(pb + 24 * NDIM, bb + 3072);
            glds16(ga0 + s * BK, a);
            glds16(ga1 + s * BK, a + 1024);
        };
#pragma unroll
        for (int s = 0; s < NS - 1; ++s) issue(s, s);
        int buf = NS - 1;
        for (int s = 0; s < NSTEP - (NS - 1); ++s) {
            WAIT_V(24);
            BAR();
            issue(s + NS - 1, buf);
            buf = (buf == NS - 1) ? 0 : buf + 1;
        }
        WAIT_V(24); BAR();
        WAIT_V(18); BAR();
        WAIT_V(12); BAR();
        WAIT_V(6); BAR();
        WAIT_V(0); BAR();
        return;
    }

    const int cw = wv;
    STAMP_AT(0);
    unsigned call = 0;
    unsigned* cnt = (unsigned*)(P.ws[rank] + WS_CNT) + b;
    if (cw == 0) call = __hip_atomic_load(cnt, __ATOMIC_RELAXED, __HIP_MEMORY_SCOPE_AGENT) + 1u;
    const float scale = P.sx[0] * P.sw[0];

    const int g = lane >> 5, r32 = lane & 31;
    int aoff[4];
#pragma unroll
    for (int j = 0; j < 4; ++j) {
        const int c = 4 * (j >> 1) + 2 * g + (j & 1);
        aoff[j] = r32 * 128 + ((c ^ ((r32 >> 1) & 7)) << 4);
    }
    const int q = (lane & 15) >> 1, p = lane & 1, cg = (lane >> 4) & 1;
    const int boff = BM * BK + (32 * g + q) * 128 + (((2 * cw + cg) ^ (((q >> 1) & 3) << 1)) << 4) + p * 8;

    f32x16 acc0, acc1;
#pragma unroll
    for (int i = 0; i < 16; ++i) { acc0[i] = 0.f; acc1[i] = 0.f; }

    i32x8 xa[2][2][2];
    i32x8 wb[2][2];
    auto rd = [&](int set, int buf) {
        const unsigned char* base = smem + buf * STAGE;
#pragma unroll
        for (int ks = 0; ks < 2; ++ks) {
#pragma unroll
            for (int i = 0; i < 4; ++i) {
                i32x2 t = __builtin_amdgcn_ds_read_tr8_b64_v2i32((LAS i32x2*)(base + boff + (64 * ks + 8 * i) * 128));
                wb[set][ks][2 * i] = t.x; wb[set][ks][2 * i + 1] = t.y;
            }
#pragma unroll
            for (int mt = 0; mt < 2; ++mt) {
#pragma unroll
                for (int h = 0; h < 2; ++h) {
                    i32x4 t = *(const i32x4*)(base + aoff[2 * ks + h] + mt * 4096);
                    xa[set][ks][mt][4 * h] = t.x; xa[set][ks][mt][4 * h + 1] = t.y;
                    xa[set][ks][mt][4 * h + 2] = t.z; xa[set][ks][mt][4 * h + 3] = t.w;
                }
            }
        }
    };
    auto mm = [&](int set) {
#pragma unroll
        for (int ks = 0; ks < 2; ++ks) {
            acc0 = __builtin_amdgcn_mfma_scale_f32_32x32x64_f8f6f4(wb[set][ks], xa[set][ks][0], acc0, 1, 0, 0, 0, 0, 0);
            acc1 = __builtin_amdgcn_mfma_scale_f32_32x32x64_f8f6f4(wb[set][ks], xa[set][ks][1], acc1, 1, 0, 0, 0, 0, 0);
        }
    };
    int buf = 0;
    BAR();
    STAMP_AT(1);
    rd(0, buf); buf = 1;
    SCHED();
    WAIT_L0();
    SCHED();
    for (int s = 1; s < NSTEP - 1; s += 2) {
        BAR();
        rd(1, buf); buf = (buf == NS - 1) ? 0 : buf + 1;
        SCHED();
        mm(0);
        SCHED();
        WAIT_L0();
        SCHED();
        BAR();
        rd(0, buf); buf = (buf == NS - 1) ? 0 : buf + 1;
        SCHED();
        mm(1);
        SCHED();
        WAIT_L0();
        SCHED();
    }
    BAR();
    rd(1, buf);
    SCHED();
    mm(0);
    SCHED();
    mm(1);

    STAMP_AT(2);
    unsigned char* E = smem + 2 * STAGE + cw * 8192;
    char* obase = (char*)P.out[dst] + ((size_t)(rank * MLOC + rb * BM) * NLOC + (cb & 3) * BN + cw * 32) * 4;
    const int r8 = lane >> 3, c8 = lane & 7;
#pragma unroll
    for (int mt = 0; mt < 2; ++mt) {
#pragma unroll
        for (int qq = 0; qq < 4; ++qq) {
            f32x4 v;
#pragma unroll
            for (int e = 0; e < 4; ++e) v[e] = (mt ? acc1[4 * qq + e] : acc0[4 * qq + e]) * scale;
            *(f32x4*)(E + (mt * 32 + r32) * 128 + (((2 * qq + g) ^ (r32 & 7)) << 4)) = v;
        }
        asm volatile("" ::: "memory");
#pragma unroll
        for (int j = 0; j < 4; ++j) {
            const int row = mt * 32 + j * 8 + r8;
            f32x4 v = *(const f32x4*)(E + row * 128 + ((c8 ^ r8) << 4));
            st16_sys(obase + (size_t)row * (NLOC * 4) + c8 * 16, v);
        }
    }
    STAMP_AT(3);
    WAIT_V(0);
    BAR();
    STAMP_AT(4);
    if (cw != 0) return;
    call = __builtin_amdgcn_readfirstlane(call);
    if (lane == 0) {
        if (dst != rank)
            __hip_atomic_store((unsigned*)(P.ws[dst] + WS_FLAG) + rank * 32 + ti, call, __ATOMIC_RELAXED, __HIP_MEMORY_SCOPE_SYSTEM);
        __hip_atomic_store(cnt, call, __ATOMIC_RELAXED, __HIP_MEMORY_SCOPE_AGENT);
    }
    if (dst == rank && ti == 0) {
        unsigned* fl = (unsigned*)(P.ws[rank] + WS_FLAG) + lane * 4;
        unsigned* tmo = (unsigned*)(P.ws[rank] + WS_TMO);
        const bool mine = (lane >> 3) == rank;
        const unsigned long long t0 = __builtin_amdgcn_s_memrealtime();
        for (;;) {
            const unsigned f0 = __hip_atomic_load(fl + 0, __ATOMIC_RELAXED, __HIP_MEMORY_SCOPE_SYSTEM);
            const unsigned f1 = __hip_atomic_load(fl + 1, __ATOMIC_RELAXED, __HIP_MEMORY_SCOPE_SYSTEM);
            const unsigned f2 = __hip_atomic_load(fl + 2, __ATOMIC_RELAXED, __HIP_MEMORY_SCOPE_SYSTEM);
            const unsigned f3 = __hip_atomic_load(fl + 3, __ATOMIC_RELAXED, __HIP_MEMORY_SCOPE_SYSTEM);
            const bool ok = mine || (f0 >= call && f1 >= call && f2 >= call && f3 >= call);
            if (__all(ok)) break;
            if (__builtin_amdgcn_s_memrealtime() - t0 > DEADLINE_TICKS) {
                if (lane == 0) {
                    __hip_atomic_store(tmo, call, __ATOMIC_RELAXED, __HIP_MEMORY_SCOPE_AGENT);
                    __hip_atomic_store((float*)obase, __builtin_nanf(""), __ATOMIC_RELAXED, __HIP_MEMORY_SCOPE_SYSTEM);
                }
                break;
            }
            __builtin_amdgcn_s_sleep(1);
        }
        STAMP_AT(5);
    }
}

extern "C" void kernel_launch(int rank, int nranks, ncclComm_t comm,
                              hipStream_t stream,
                              void* const* d_in, const int* in_sizes, int n_in,
                              void* d_out, int out_size,
                              const int* in_dtypes, int out_dtype,
                              void* const* d_out_all,
                              void* const* d_ws_all, size_t ws_bytes) {
    if (nranks != NR || (unsigned)rank >= (unsigned)NR || n_in < 4 || ws_bytes < WS_NEED ||
        in_sizes[0] != MLOC * KDIM || in_sizes[1] != KDIM * NDIM || out_size != NR * MLOC * NLOC) {
        fprintf(stderr, "kernel_launch: unexpected problem shape\n");
        return;
    }
    static bool attr_set = false;
    if (!attr_set) {
        HIP_CHECK(hipFuncSetAttribute((const void*)gemm_a2a, hipFuncAttributeMaxDynamicSharedMemorySize, LDS_BYTES));
        attr_set = true;
    }
    Params P;
    P.x = (const uint8_t*)d_in[0];
    P.w = (const uint8_t*)d_in[1];
    P.sx = (const float*)d_in[2];
    P.sw = (const float*)d_in[3];
    for (int p = 0; p < NR; ++p) { P.out[p] = (float*)d_out_all[p]; P.ws[p] = (char*)d_ws_all[p]; }
    P.rank = rank;
    hipLaunchKernelGGL(gemm_a2a, dim3(NWG), dim3(512), LDS_BYTES, stream, P);
    HIP_CHECK(hipGetLastError());
}
```

```cpp
#include <hip/hip_runtime.h>
#include <rccl/rccl.h>
#include <cstdio>
#include <cstdint>

#define HIP_CHECK(expr) do { hipError_t _e = (expr); if (_e != hipSuccess) { \
    fprintf(stderr, "HIP error %s at %s:%d\n", hipGetErrorString(_e), __FILE__, __LINE__); \
    return; } } while (0)

#define XFB 4
#define NPUSH 4
#define PUSH_INFL (8 / NPUSH)

#define LATE_FENCE 0
#define PFORD 1
#define NOLATE 0

typedef int i32x8 __attribute__((ext_vector_type(8)));
typedef int i32x4 __attribute__((ext_vector_type(4)));
typedef int i32x2 __attribute__((ext_vector_type(2)));
typedef float f32x4 __attribute__((ext_vector_type(4)));
#define LAS __attribute__((address_space(3)))
#define GAS __attribute__((address_space(1)))
#define CAS __attribute__((address_space(4)))

constexpr int NR = 8;
constexpr int KDIM = 4096, NDIM = 4096, MLOC = 512, NLOC = 512;
constexpr int NBLK = MLOC / 16;
constexpr int SBLK = NBLK - XFB;
constexpr int SB0 = (SBLK + 1) / 2;
constexpr int WAVE_LDS = 20480;
constexpr int LDS_BYTES = 8 * WAVE_LDS;
constexpr int NWG = 256;

constexpr size_t WS_FLAG = 0;
constexpr size_t WS_XFLAG = 2048;
constexpr size_t WS_CNT = 4096;
constexpr size_t WS_TMO = 8192;
constexpr size_t WS_XBUF = 1u << 20;
constexpr size_t XSLOT = (size_t)XFB * 16 * KDIM;
constexpr size_t WS_NEED = WS_XBUF + 2 * NR * XSLOT;
constexpr unsigned long long DEADLINE_TICKS = 50000000ull;
#define STAMP_AT(k) do {} while (0)
#define FINE_AT(k) do {} while (0)
#define STAMP_OUT() do {} while (0)

struct Params {
    const uint8_t* x;
    const uint8_t* w;
    char* myws;
    float* myout;
    const float* sx;
    const float* sw;
    int rank;
    int pad;
    float* out[NR];
    char* ws[NR];
};

__device__ __forceinline__ void st16_sys(void* p, f32x4 v) {
    asm volatile("global_store_dwordx4 %0, %1, off sc0 sc1\n\ts_nop 1" :: "v"(p), "v"(v) : "memory");
}
#define STR_(x) #x
#define STR(x) STR_(x)
#define WAIT_V(n) asm volatile("s_waitcnt vmcnt(" STR(n) ")" ::: "memory")
#define WAIT_VN(n) asm volatile("s_waitcnt vmcnt(%0)" :: "n"(n) : "memory")
#define WAIT_L0() do { __builtin_amdgcn_s_waitcnt(0xC07F); asm volatile("" ::: "memory"); } while (0)
#define BAR() asm volatile("s_barrier" ::: "memory")
#define SCHED() __builtin_amdgcn_sched_barrier(0)
#define RLX_SYS __ATOMIC_RELAXED, __HIP_MEMORY_SCOPE_SYSTEM
#define RLX_AGT __ATOMIC_RELAXED, __HIP_MEMORY_SCOPE_AGENT

__device__ __forceinline__ void glds16(const uint8_t* base, unsigned voff, unsigned lds) {
    asm volatile("s_mov_b32 m0, %2\n\ts_nop 0\n\tglobal_load_lds_dwordx4 %0, %1" :: "v"(voff), "s"(base), "s"(lds) : "memory");
}

__device__ __forceinline__ void glds16_sys(const uint8_t* base, unsigned voff, unsigned lds) {
    asm volatile("s_mov_b32 m0, %2\n\ts_nop 0\n\tglobal_load_lds_dwordx4 %0, %1 sc0 sc1" :: "v"(voff), "s"(base), "s"(lds) : "memory");
}

__global__ void __launch_bounds__(512, 2) gemm_a2a(Params P) {
    extern __shared__ __attribute__((aligned(16))) unsigned char smem[];
    const int lane = threadIdx.x & 63;
    const int wv = __builtin_amdgcn_readfirstlane(threadIdx.x >> 6);
    const int b = blockIdx.x;
    const int lab = b & 7, ti = b >> 3;
    int rank = P.rank;
    GAS char* myws = (GAS char*)P.myws;
    const uint8_t* gx = P.x;
    const uint8_t* gw = P.w;
    asm volatile("" : "+s"(rank), "+s"(myws), "+s"(gx), "+s"(gw));
    GAS unsigned* const cnt = (GAS unsigned*)(myws + WS_CNT) + b;
    GAS unsigned* const tmo = (GAS unsigned*)(myws + WS_TMO);
    STAMP_AT(0);

    const int slice = ti & 7, grp = (ti >> 3) & 1;
    const bool remote = lab != rank;
    const bool aux = ti >= 16;
    const unsigned lbase = (unsigned)(uintptr_t)(LAS unsigned char*)smem + wv * WAVE_LDS;
    const int nb0 = remote ? SB0 : NBLK / 2;
    const int nb1 = remote ? SBLK - SB0 : NBLK / 2;
    unsigned call = 0;

    if (aux) {
        const int q = ti - 16;
        unsigned prev;
        if (q >= 16 - NPUSH) {
            asm volatile("global_load_dword %0, %1, %2 sc1" : "=v"(prev) : "v"(0u), "s"(cnt) : "memory");
            if (remote) {
                constexpr int NX = XFB * 8 / NPUSH;
                const unsigned xoff = ((q - (16 - NPUSH)) * 8 + wv) * (NX * 1024) + lane * 16;
                const uint8_t* src = gx + (size_t)SBLK * 16 * KDIM;
                f32x4 v[NX];
#pragma unroll
                for (int i = 0; i < NX; ++i)
                    asm volatile("global_load_dwordx4 %0, %1, %2" : "=v"(v[i]) : "v"(xoff + i * 1024), "s"(src) : "memory");
                char* const pws = P.ws[lab];
                WAIT_VN(NX);
                asm volatile("" : "+v"(prev));
                call = __builtin_amdgcn_readfirstlane(prev) + 1u;
                char* d = pws + WS_XBUF + ((call & 1u) * NR + rank) * XSLOT + xoff;
#pragma unroll
                for (int i = 0; i < NX; ++i) {
                    if (i < PUSH_INFL) WAIT_VN(NX - 1); else WAIT_VN(PUSH_INFL - 1);
                    asm volatile("" : "+v"(v[i]));
                    st16_sys(d + i * 1024, v[i]);
                    if (i == 0) STAMP_AT(6);
                }
                WAIT_V(0);
                BAR();
                STAMP_AT(7);
                if (threadIdx.x == 0)
                    __hip_atomic_store((unsigned*)(pws + WS_XFLAG) + rank * 32 + (q - (16 - NPUSH)), call, RLX_SYS);
            }
        } else {
            constexpr int NPW = (16 - NPUSH) * 8;
            const unsigned vl = lane * 16;
            const unsigned vp = (lane >> 5) * NDIM + (lane & 31) * 16;
            const int u = q * 8 + wv;
            auto pfA = [&](int m) {
                const int i = m >> 7, hi = (m >> 6) & 1;
                const int blk = hi ? nb0 + (i < nb1 ? i : nb1 - 1) : i;
                glds16(gx + (size_t)blk * 16 * KDIM + (m & 63) * 1024, vl, lbase);
            };
            auto pfB = [&](const uint8_t* base, int m) {
                const int row = ((m >> 6) & 3) * 1024 + (m >> 8) * 128 + (m & 63) * 2;
                glds16(base + (size_t)row * NDIM, vp, lbase);
            };
            const uint8_t* wl = gw + lab * NLOC;
#pragma unroll 1
            for (int m = u; m < 128; m += NPW) pfA(m);
#pragma unroll 1
            for (int m = NPW - 1 - u; m < 2048; m += NPW) { pfB(wl, m); WAIT_V(12); }
            asm volatile("global_load_dword %0, %1, %2 sc1" : "=v"(prev) : "v"(0u), "s"(cnt) : "memory");
            BAR();
#pragma unroll 1
            for (int m = 128 + u; m < nb0 * 128; m += NPW) { pfA(m); WAIT_V(12); }
            if (remote && !NOLATE) {
                BAR();
                const uint8_t* wo = gw + rank * NLOC;
#pragma unroll 1
                for (int m = u; m < 2048; m += NPW) { pfB(wo, m); WAIT_V(12); }
            }
        }
        WAIT_V(0);
        asm volatile("" : "+v"(prev));
        call = __builtin_amdgcn_readfirstlane(prev) + 1u;
    }

    const bool late = aux;
    const int dst = late ? rank : lab;
    int nblk, blk0;
    if (late) { nblk = grp ? XFB / 2 : (XFB + 1) / 2; blk0 = grp ? (XFB + 1) / 2 : 0; }
    else { nblk = grp ? nb1 : nb0; blk0 = grp ? nb0 : 0; }
    const uint8_t* abase = gx + (size_t)blk0 * 16 * KDIM;
    const int orow0 = late ? lab * MLOC + (SBLK + blk0) * 16 : rank * MLOC + blk0 * 16;

    if (!aux || (remote && !NOLATE)) {
        bool bad = false;
        if (late) {
            if (wv == 0) {
                GAS unsigned* fl = (GAS unsigned*)(myws + WS_XFLAG) + lab * 32 + (lane & (NPUSH - 1));
                const unsigned long long t0 = __builtin_amdgcn_s_memrealtime();
                while (!__all(__hip_atomic_load(fl, RLX_SYS) >= call)) {
                    if (__hip_atomic_load(tmo, RLX_AGT) == call || __builtin_amdgcn_s_memrealtime() - t0 > DEADLINE_TICKS) {
                        __hip_atomic_store(tmo, call, RLX_AGT);
                        break;
                    }
                    __builtin_amdgcn_s_sleep(8);
                }
                WAIT_V(0);
            }
            BAR();
            bad = __builtin_amdgcn_readfirstlane(__hip_atomic_load(tmo, RLX_AGT)) == call;
            abase = (const uint8_t*)(char*)myws + WS_XBUF + ((call & 1u) * NR + lab) * XSLOT + (size_t)blk0 * 16 * KDIM;
            STAMP_AT(8);
        }

        unsigned char* const lw = smem + wv * WAVE_LDS;
        const int r8 = lane >> 3, c8 = lane & 7;
        const unsigned va0 = r8 * KDIM + ((c8 ^ (r8 >> 1)) << 4);
        const unsigned va1 = (8 + r8) * KDIM + ((c8 ^ (4 + (r8 >> 1))) << 4);
        const uint8_t* aw = abase + wv * 128;
        auto loadA = [&](int blk, int half, int so) {
            const bool in = blk < nblk;
            const uint8_t* p = aw + (size_t)(in ? blk : 0) * 16 * KDIM + half * 2048;
            const unsigned o0 = in ? va0 : 0u, o1 = in ? va1 : 0u;
            const unsigned l = lbase + so;
            if (late && !LATE_FENCE) {
                glds16_sys(p, o0, l); glds16_sys(p, o1, l + 1024);
                glds16_sys(p + 1024, o0, l + 2048); glds16_sys(p + 1024, o1, l + 3072);
            } else {
                glds16(p, o0, l); glds16(p, o1, l + 1024);
                glds16(p + 1024, o0, l + 2048); glds16(p + 1024, o1, l + 3072);
            }
        };
        const int r16 = lane >> 2, c4 = lane & 3;
        const unsigned vb0 = r16 * NDIM + ((c4 ^ ((r16 >> 2) & 1)) << 4);
        const unsigned vb1 = (16 + r16) * NDIM + ((c4 ^ (((r16 >> 2) & 1) | 2)) << 4);
        const uint8_t* bw = gw + (size_t)wv * 128 * NDIM + dst * NLOC + slice * 64;
        auto loadB = [&](int gI, int slot) {
            const uint8_t* p = bw + (size_t)((gI >> 1) * 1024 + (gI & 1) * 64) * NDIM;
            const unsigned l = lbase + slot * 4096;
            glds16(p, vb0, l); glds16(p, vb1, l + 1024);
            glds16(p + 32 * NDIM, vb0, l + 2048); glds16(p + 32 * NDIM, vb1, l + 3072);
        };
        FINE_AT(1);
        loadA(0, 0, 0); loadA(0, 1, 4096);
        loadB(0, 2); loadB(1, 3); loadB(2, 4);
        FINE_AT(2);

        float scx, scw, scale;
        asm volatile("s_load_dword %0, %2, 0x0\n\ts_load_dword %1, %3, 0x0" : "=&s"(scx), "=&s"(scw) : "s"(P.sx), "s"(P.sw) : "memory");
        const int i16 = lane & 15, g = lane >> 4;
        const int ra0 = i16 * 128 + ((g ^ (i16 >> 1)) << 4);
        const int ra1 = i16 * 128 + (((4 + g) ^ (i16 >> 1)) << 4);
        i32x8 xa[4];
        auto readA = [&](int half, int so) {
#pragma unroll
            for (int k2 = 0; k2 < 2; ++k2) {
                const unsigned char* p = lw + so + k2 * 2048;
                const i32x4 t0 = *(const i32x4*)(p + ra0), t1 = *(const i32x4*)(p + ra1);
                i32x8& d = xa[half * 2 + k2];
                d[0] = t0.x; d[1] = t0.y; d[2] = t0.z; d[3] = t0.w; d[4] = t1.x; d[5] = t1.y; d[6] = t1.z; d[7] = t1.w;
            }
        };
        const int q = i16 >> 1, pp = lane & 1;
        const int fB = ((q >> 2) & 1) | ((g & 1) << 1);
        int rb[4];
#pragma unroll
        for (int t = 0; t < 4; ++t) rb[t] = g * 1024 + q * 64 + ((t ^ fB) << 4) + pp * 8;
        i32x8 wb[4][4];
        auto readB = [&](int gI, int slot) {
            const int ks = gI >> 1, h = gI & 1;
#pragma unroll
            for (int t = 0; t < 4; ++t) {
#pragma unroll
                for (int r = 0; r < 2; ++r) {
                    const i32x2 v = __builtin_amdgcn_ds_read_tr8_b64_v2i32((LAS i32x2*)(lw + slot * 4096 + rb[t] + r * 512));
                    wb[ks][t][4 * h + 2 * r] = v.x; wb[ks][t][4 * h + 2 * r + 1] = v.y;
                }
            }
        };
        f32x4 acc[4];
        auto mm = [&](int ks) {
#pragma unroll
            for (int t = 0; t < 4; ++t) {
                const f32x4 c = ks == 0 ? f32x4{0.f, 0.f, 0.f, 0.f} : acc[t];
                acc[t] = __builtin_amdgcn_mfma_scale_f32_16x16x128_f8f6f4(wb[ks][t], xa[ks], c, 1, 0, 0, 0, 0, 0);
            }
        };
        unsigned char* const pw = lw + i16 * 256;
        const int prow = 2 * wv + (g & 1);
        const unsigned char* const pr = smem + (g >> 1) * 4 * WAVE_LDS + prow * 256 + ((i16 ^ (prow & 7)) << 4);
        char* optr = (char*)(late ? P.myout : P.out[lab]) + ((size_t)(orow0 + prow) * NLOC + slice * 64 + i16 * 4) * 4;
        f32x4 ps[4];
        auto writeP = [&](int so) {
#pragma unroll
            for (int t = 0; t < 4; ++t) *(f32x4*)(pw + so + (((4 * t + g) ^ (i16 & 7)) << 4)) = acc[t];
        };
        auto readP = [&](int so) {
#pragma unroll
            for (int j = 0; j < 4; ++j) ps[j] = *(const f32x4*)(pr + so + j * WAVE_LDS);
        };
        auto emit = [&]() {
            f32x4 s = (ps[0] + ps[1]) + (ps[2] + ps[3]);
#pragma unroll
            for (int e = 0; e < 4; ++e) {
                const auto r = __builtin_amdgcn_permlane32_swap(__float_as_uint(s[e]), __float_as_uint(s[e]), false, false);
                s[e] = (__uint_as_float(r[0]) + __uint_as_float(r[1])) * scale;
            }
            if (lane < 32) st16_sys(optr, s);
            optr += 16 * NLOC * 4;
        };

        WAIT_V(16); FINE_AT(3); readA(0, 0); WAIT_L0(); loadB(3, 0);
        WAIT_V(16); readA(1, 4096); WAIT_L0(); loadB(4, 1);
        asm volatile("" : "+s"(scx), "+s"(scw));
        scale = bad ? __builtin_nanf("") : scx * scw;
        WAIT_V(16); FINE_AT(4); readB(0, 2); WAIT_L0(); loadB(5, 2);
        WAIT_V(16); readB(1, 3); WAIT_L0(); loadB(6, 3); SCHED(); mm(0); SCHED();
        WAIT_V(16); readB(2, 4); WAIT_L0(); loadB(7, 4);
        WAIT_V(16); FINE_AT(6); readB(3, 0); SCHED(); mm(1); SCHED();
        WAIT_V(12); readB(4, 1);
        WAIT_V(8); readB(5, 2); SCHED(); mm(2); SCHED();
        WAIT_V(4); readB(6, 3);
        WAIT_V(0); FINE_AT(7); readB(7, 4); WAIT_L0(); SCHED(); mm(3); SCHED();
        writeP(4 * 4096);
        WAIT_L0();
        BAR();
        readP(4 * 4096);
        loadA(1, 0, 0); loadA(1, 1, 4096); loadA(2, 0, 2 * 4096);
        emit();
        FINE_AT(8);
        if (!late) { if (remote) STAMP_AT(1); else STAMP_AT(2); }

        int cur0 = 0, cur1 = 4096, nxt0 = 2 * 4096, fre = 3 * 4096, prv = 4 * 4096;
#pragma unroll 1
        for (int i = 1; i < nblk; ++i) {
            if (i == 6) FINE_AT(9);
            loadA(i + 1, 1, fre);
            if (i > 1) readP(prv);
            if (i == 2) WAIT_V(8); else WAIT_V(9);
            if (i == 6) FINE_AT(10);
            readA(0, cur0); readA(1, cur1);
            WAIT_L0();
            SCHED();
            mm(0);
            SCHED();
            loadA(i + 2, 0, cur1);
            SCHED();
            mm(1);
            SCHED();
            if (i > 1) emit();
            SCHED();
            mm(2); mm(3);
            SCHED();
            if (i == 6) FINE_AT(11);
            writeP(cur0);
            WAIT_L0();
            if (i == 6) FINE_AT(12);
            BAR();
            if (i == 6) FINE_AT(13);
            const int t = cur0; cur0 = nxt0; nxt0 = cur1; cur1 = fre; fre = prv; prv = t;
        }
        if (nblk > 1) { readP(prv); emit(); }
        if (remote && !late) STAMP_AT(3);
        if (!aux) call = __builtin_amdgcn_readfirstlane(__hip_atomic_load(cnt, RLX_AGT)) + 1u;
    }
    WAIT_V(0);
    BAR();
    if (late) STAMP_AT(9); else if (remote) STAMP_AT(4); else STAMP_AT(10);
    if (wv != 0) return;
    if (lane == 0) {
        if (dst != rank) __hip_atomic_store((unsigned*)(P.ws[lab] + WS_FLAG) + rank * 32 + ti, call, RLX_SYS);
        __hip_atomic_store(cnt, call, RLX_AGT);
    }
    if (!remote && ti == 31) {
        GAS unsigned* fl = (GAS unsigned*)(myws + WS_FLAG) + (lane >> 3) * 32 + (lane & 7) * 2;
        const bool skip = (lane >> 3) == rank;
        const unsigned long long t0 = __builtin_amdgcn_s_memrealtime();
        for (;;) {
            const unsigned long long f = __hip_atomic_load((GAS unsigned long long*)fl, RLX_SYS);
            const bool ok = skip || ((unsigned)f >= call && (unsigned)(f >> 32) >= call);
            if (__all(ok)) break;
            if (__builtin_amdgcn_s_memrealtime() - t0 > DEADLINE_TICKS) {
                if (lane == 0) {
                    __hip_atomic_store(tmo, call, RLX_AGT);
                    __hip_atomic_store(P.myout + (size_t)rank * MLOC * NLOC, __builtin_nanf(""), RLX_SYS);
                }
                break;
            }
            __builtin_amdgcn_s_sleep(1);
        }
        STAMP_AT(5);
    }
    STAMP_OUT();
}

extern "C" void kernel_launch(int rank, int nranks, ncclComm_t comm,
                              hipStream_t stream,
                              void* const* d_in, const int* in_sizes, int n_in,
                              void* d_out, int out_size,
                              const int* in_dtypes, int out_dtype,
                              void* const* d_out_all,
                              void* const* d_ws_all, size_t ws_bytes) {
    if (nranks != NR || (unsigned)rank >= (unsigned)NR || n_in < 4 || ws_bytes < WS_NEED ||
        in_sizes[0] != MLOC * KDIM || in_sizes[1] != KDIM * NDIM || out_size != NR * MLOC * NLOC) {
        fprintf(stderr, "kernel_launch: unexpected problem shape\n");
        return;
    }
    static bool attr_set = false;
    if (!attr_set) {
        HIP_CHECK(hipFuncSetAttribute((const void*)gemm_a2a, hipFuncAttributeMaxDynamicSharedMemorySize, LDS_BYTES));
        attr_set = true;
    }
    Params P;
    P.x = (const uint8_t*)d_in[0];
    P.w = (const uint8_t*)d_in[1];
    P.sx = (const float*)d_in[2];
    P.sw = (const float*)d_in[3];
    for (int p = 0; p < NR; ++p) { P.out[p] = (float*)d_out_all[p]; P.ws[p] = (char*)d_ws_all[p]; }
    P.myws = P.ws[rank];
    P.myout = P.out[rank];
    P.rank = rank;
    P.pad = 0;
    hipLaunchKernelGGL(gemm_a2a, dim3(NWG), dim3(512), LDS_BYTES, stream, P);
    HIP_CHECK(hipGetLastError());
}
```

```cpp
#include <hip/hip_runtime.h>
#include <rccl/rccl.h>
#include <cstdio>
#include <cstdint>

#define HIP_CHECK(expr) do { hipError_t _e = (expr); if (_e != hipSuccess) { \
    fprintf(stderr, "HIP error %s at %s:%d\n", hipGetErrorString(_e), __FILE__, __LINE__); \
    return; } } while (0)

#define XFB 4
#define NPUSH 4
#define PUSH_INFL (8 / NPUSH)

#define LATE_FENCE 0
#define PFORD 1
#define NOLATE 0

#define L1EARLY 1
#define NOSTORE 0
#define NOLOAD 0
#define XPRIO 1
#define PFGATE 550
#define PFINFL 12
#define PFINFL2 12
#define NG2 4
#define NL23 12
#define YRD 2
#define PUSH_WIN 8
#define YBAR 1
#define PRIO 0
#define SETPRIO(n) do {} while (0)

typedef int i32x8 __attribute__((ext_vector_type(8)));
typedef int i32x4 __attribute__((ext_vector_type(4)));
typedef int i32x2 __attribute__((ext_vector_type(2)));
typedef float f32x4 __attribute__((ext_vector_type(4)));
typedef unsigned u32x4 __attribute__((ext_vector_type(4)));
#define LAS __attribute__((address_space(3)))
#define GAS __attribute__((address_space(1)))
#define CAS __attribute__((address_space(4)))

constexpr int NR = 8;
constexpr int KDIM = 4096, NDIM = 4096, MLOC = 512, NLOC = 512;
constexpr int NBLK = MLOC / 16;
constexpr int SBLK = NBLK - XFB;
constexpr int SB0 = (SBLK + 1) / 2;
constexpr int WAVE_LDS = 20480;
constexpr int LDS_BYTES = 8 * WAVE_LDS;
constexpr int NWG = 256;
static_assert(XFB >= 2 && XFB <= 6 && NL23 >= 2 && NL23 % 2 == 0, "block split");

constexpr size_t WS_FLAG = 0;
constexpr size_t WS_XFLAG = 2048;
constexpr size_t WS_CNT = 4096;
constexpr size_t WS_TMO = 8192;
constexpr size_t WS_XBUF = 1u << 20;
constexpr size_t XSLOT = (size_t)XFB * 16 * KDIM;
constexpr size_t WS_NEED = WS_XBUF + 2 * NR * XSLOT;
constexpr unsigned long long DEADLINE_TICKS = 50000000ull;
#define STAMP_AT(k) do {} while (0)
#define FINE_AT(k) do {} while (0)
#define STAMP_OUT() do {} while (0)

#define CYC_AT(k) do {} while (0)
#define CYC7() do {} while (0)
#define SOLO 0

struct Params {
    const uint8_t* x;
    const uint8_t* w;
    char* myws;
    float* myout;
    const float* sx;
    const float* sw;
    int rank;
    int pad;
    float* out[NR];
    char* ws[NR];
};

__device__ __forceinline__ void st16_sys(void* p, f32x4 v) {
    asm volatile("global_store_dwordx4 %0, %1, off sc0 sc1\n\ts_nop 1" :: "v"(p), "v"(v) : "memory");
}
#define PAD ""
#define STR_(x) #x
#define STR(x) STR_(x)
#define WAIT_V(n) asm volatile("s_waitcnt vmcnt(" STR(n) ")" ::: "memory")
#define WAIT_VN(n) asm volatile("s_waitcnt vmcnt(%0)" :: "n"(n) : "memory")
#define WAIT_L0() do { __builtin_amdgcn_s_waitcnt(0xC07F); asm volatile("" ::: "memory"); } while (0)
#define BAR() asm volatile("s_barrier" ::: "memory")
#define SCHED() __builtin_amdgcn_sched_barrier(0)
#define RLX_SYS __ATOMIC_RELAXED, __HIP_MEMORY_SCOPE_SYSTEM
#define RLX_AGT __ATOMIC_RELAXED, __HIP_MEMORY_SCOPE_AGENT

__device__ __forceinline__ void glds16(const uint8_t* base, unsigned voff, unsigned lds) {
    asm volatile(PAD "s_mov_b32 m0, %2\n\ts_nop 0\n\tglobal_load_lds_dwordx4 %0, %1" :: "v"(voff), "s"(base), "s"(lds) : "memory");
}

__device__ __forceinline__ void glds16_sys(const uint8_t* base, unsigned voff, unsigned lds) {
    asm volatile(PAD "s_mov_b32 m0, %2\n\ts_nop 0\n\tglobal_load_lds_dwordx4 %0, %1 sc0 sc1" :: "v"(voff), "s"(base), "s"(lds) : "memory");
}

template <int I> struct IC { static constexpr int value = I; };
template <int N, int I = 0, class F> __device__ __forceinline__ void static_for(F&& f) {
    if constexpr (I < N) { f(IC<I>{}); static_for<N, I + 1>(f); }
}
constexpr int push_wait(int i, int n, int w, int f) {
    int c = 0;
    if (i < f) { c = (w - 1 - i) + i; for (int k = 0; k < i; ++k) c += (w + k < n); return c; }
    for (int k = i - f; k < i; ++k) c += (w + k < n) + (k > i - f);
    return c;
}

__global__ void __launch_bounds__(512, 2) gemm_a2a(Params P) {
    extern __shared__ __attribute__((aligned(16))) unsigned char smem[];
    const int lane = threadIdx.x & 63;
    const int wv = __builtin_amdgcn_readfirstlane(threadIdx.x >> 6);
    const int b = blockIdx.x;
    const int lab = b & 7, ti = b >> 3;
    int rank = P.rank;
    GAS char* myws = (GAS char*)P.myws;
    const uint8_t* gx = P.x;
    const uint8_t* gw = P.w;
    asm volatile("" : "+s"(rank), "+s"(myws), "+s"(gx), "+s"(gw));
    GAS unsigned* const cnt = (GAS unsigned*)(myws + WS_CNT) + b;
    GAS unsigned* const tmo = (GAS unsigned*)(myws + WS_TMO);
    STAMP_AT(0);

    const int slice = ti & 7, grp = ti >> 3;
    const bool remote = lab != rank;
    const bool aux = ti >= 16;
    const bool late = remote && aux;
    const unsigned lbase = (unsigned)(uintptr_t)(LAS unsigned char*)smem + wv * WAVE_LDS;
    unsigned call = 0;

    if (aux) {
        const int q = ti - 16;
        const unsigned long long tstart = __builtin_amdgcn_s_memrealtime();
        const unsigned vp = (lane >> 5) * NDIM + (lane & 31) * 16;
        auto pfB = [&](const uint8_t* base, int m) {
            const int row = ((m >> 6) & 3) * 1024 + (m >> 8) * 128 + (m & 63) * 2;
            glds16(base + (size_t)row * NDIM, vp, lbase);
        };
        unsigned prev;
        if (q >= 16 - NPUSH) {
            asm volatile("s_nop 4\n\tglobal_load_dword %0, %1, %2 sc1" : "=v"(prev) : "v"(0u), "s"(cnt) : "memory");
            if (remote) {
                constexpr int NX = XFB * 8 / NPUSH;
                const unsigned xoff = ((q - (16 - NPUSH)) * 8 + wv) * (NX * 1024) + lane * 16;
                const uint8_t* src = gx + (size_t)SBLK * 16 * KDIM;
                constexpr int PW = NX < PUSH_WIN ? NX : PUSH_WIN;
                static_assert(PW == NX || PW > PUSH_INFL, "window");
                f32x4 v[NX];
#pragma unroll
                for (int i = 0; i < PW; ++i)
                    asm volatile(PAD "global_load_dwordx4 %0, %1, %2" : "=v"(v[i]) : "v"(xoff + i * 1024), "s"(src) : "memory");
                char* const pws = P.ws[lab];
                WAIT_VN(PW);
                asm volatile("" : "+v"(prev));
                call = __builtin_amdgcn_readfirstlane(prev) + 1u;
                char* d = pws + WS_XBUF + ((call & 1u) * NR + rank) * XSLOT + xoff;
                static_for<NX>([&](auto ic) {
                    constexpr int i = decltype(ic)::value;
                    WAIT_VN(push_wait(i, NX, PW, PUSH_INFL));
                    asm volatile("" : "+v"(v[i]));
                    st16_sys(d + i * 1024, v[i]);
                    if constexpr (PW + i < NX)
                        asm volatile(PAD "global_load_dwordx4 %0, %1, %2" : "=v"(v[PW + i]) : "v"(xoff + (PW + i) * 1024), "s"(src) : "memory");
                    if (i == 0) STAMP_AT(6);
                });
                WAIT_V(0);
                BAR();
                STAMP_AT(7);
                if (threadIdx.x == 0)
                    __hip_atomic_store((unsigned*)(pws + WS_XFLAG) + rank * 32 + (q - (16 - NPUSH)), call, RLX_SYS);
            }
            while (__builtin_amdgcn_s_memrealtime() - tstart < PFGATE) __builtin_amdgcn_s_sleep(2);
        } else {
            constexpr int NPW = (16 - NPUSH) * 8;
            const unsigned vl = lane * 16;
            const int u = q * 8 + wv;
            const uint8_t* wl = gw + lab * NLOC;
#pragma unroll 1
            for (int m = u; m < 128; m += NPW) glds16(gx + m * 1024, vl, lbase);
#pragma unroll 1
            for (int m = NPW - 1 - u; m < 2048; m += NPW) { pfB(wl, m); WAIT_V(PFINFL); }
            asm volatile("s_nop 4\n\tglobal_load_dword %0, %1, %2 sc1" : "=v"(prev) : "v"(0u), "s"(cnt) : "memory");
            BAR();
#pragma unroll 1
            for (int m = 128 + u; m < 3 * 128; m += NPW) glds16(gx + m * 1024, vl, lbase);
            while (__builtin_amdgcn_s_memrealtime() - tstart < PFGATE) __builtin_amdgcn_s_sleep(2);
            const int mend = (remote ? SBLK : NBLK) * 64;
#pragma unroll 1
            for (int m = 3 * 128 + u; m < mend; m += NPW) { glds16(gx + m * 1024, vl, lbase); WAIT_V(PFINFL2); }
            if (remote) {
                BAR();
                const uint8_t* wo = gw + rank * NLOC;
#pragma unroll 1
                for (int m = u; m < 2048; m += NPW) { pfB(wo, m); WAIT_V(PFINFL2); }
            }
        }

        WAIT_V(0);
        asm volatile("" : "+v"(prev));
        call = __builtin_amdgcn_readfirstlane(prev) + 1u;
    }

    const int dst = late ? rank : lab;
    int nblk, blk0;
    constexpr int bstr = 2;
    {
        const int nmain = remote ? SBLK : NBLK - NL23;
        if (late) { nblk = (XFB + 1 - (grp & 1)) >> 1; blk0 = grp & 1; }
        else if (!aux) { nblk = (nmain + 1 - grp) >> 1; blk0 = grp; }
        else { nblk = NL23 / 2; blk0 = nmain + (grp & 1); }
    }
    const uint8_t* abase = gx + (size_t)blk0 * 16 * KDIM;
    const int orow0 = late ? lab * MLOC + (SBLK + blk0) * 16 : rank * MLOC + blk0 * 16;
    constexpr int astep = bstr * 16 * KDIM, ostep = bstr * 16 * NLOC * 4;

    {
        bool bad = false;
        if (late) {
            if (wv == 0) {
                GAS unsigned* fl = (GAS unsigned*)(myws + WS_XFLAG) + lab * 32 + (lane & (NPUSH - 1));
                const unsigned long long t0 = __builtin_amdgcn_s_memrealtime();
                while (!SOLO && !__all(__hip_atomic_load(fl, RLX_SYS) >= call)) {
                    if (__hip_atomic_load(tmo, RLX_AGT) == call || __builtin_amdgcn_s_memrealtime() - t0 > DEADLINE_TICKS) {
                        __hip_atomic_store(tmo, call, RLX_AGT);
                        break;
                    }
                    __builtin_amdgcn_s_sleep(8);
                }
                WAIT_V(0);
            }
            BAR();
            bad = __builtin_amdgcn_readfirstlane(__hip_atomic_load(tmo, RLX_AGT)) == call;
            abase = (const uint8_t*)(char*)myws + WS_XBUF + ((call & 1u) * NR + lab) * XSLOT + (size_t)blk0 * 16 * KDIM;
            STAMP_AT(8);
        }

        unsigned char* const lw = smem + wv * WAVE_LDS;
        const int r8 = lane >> 3, c8 = lane & 7;
        const unsigned va0 = r8 * KDIM + ((c8 ^ (r8 >> 1)) << 4);
        const unsigned va1 = (8 + r8) * KDIM + ((c8 ^ (4 + (r8 >> 1))) << 4);
        const uint8_t* aw = abase + wv * 128;
        auto loadAq = [&](int blk, int ks, int so, bool sys) {
            const bool in = blk < (NOLOAD ? 1 : nblk);
            const uint8_t* p = aw + (in ? blk : 0) * astep + ks * 1024;
            const unsigned o0 = in ? va0 : 0u, o1 = in ? va1 : 0u;
            const unsigned l = lbase + so + (ks & 1) * 2048;
            if (sys) { glds16_sys(p, o0, l); glds16_sys(p, o1, l + 1024); }
            else { glds16(p, o0, l); glds16(p, o1, l + 1024); }
        };
        auto loadA2 = [&](int blk, int so0, int so1) {
            if (late) { loadAq(blk, 0, so0, true); loadAq(blk, 1, so0, true); loadAq(blk, 2, so1, true); loadAq(blk, 3, so1, true); }
            else { loadAq(blk, 0, so0, false); loadAq(blk, 1, so0, false); loadAq(blk, 2, so1, false); loadAq(blk, 3, so1, false); }
        };
        const int r16 = lane >> 2, c4 = lane & 3;
        const unsigned vb0 = r16 * NDIM + ((c4 ^ ((r16 >> 2) & 1)) << 4);
        const unsigned vb1 = (16 + r16) * NDIM + ((c4 ^ (((r16 >> 2) & 1) | 2)) << 4);
        const uint8_t* bw = gw + (size_t)wv * 128 * NDIM + dst * NLOC + slice * 64;
        auto loadB = [&](int gI, int slot) {
            const uint8_t* p = bw + (size_t)((gI >> 1) * 1024 + (gI & 1) * 64) * NDIM;
            const unsigned l = lbase + slot * 4096;
            glds16(p, vb0, l); glds16(p, vb1, l + 1024);
            glds16(p + 32 * NDIM, vb0, l + 2048); glds16(p + 32 * NDIM, vb1, l + 3072);
        };
        FINE_AT(1);
        SETPRIO(1);
        loadA2(0, 0, 4096);
        loadB(0, 2); loadB(1, 3); loadB(2, 4);
        FINE_AT(2);

        float scx, scw, scale;
        asm volatile(PAD "s_load_dword %0, %2, 0x0\n\ts_load_dword %1, %3, 0x0" : "=&s"(scx), "=&s"(scw) : "s"(P.sx), "s"(P.sw) : "memory");
        const int i16 = lane & 15, g = lane >> 4;
        const int ra0 = i16 * 128 + ((g ^ (i16 >> 1)) << 4);
        const int ra1 = i16 * 128 + (((4 + g) ^ (i16 >> 1)) << 4);
        i32x8 xa[4];
        auto readAq = [&](int ks, int so) {
            const unsigned char* p = lw + so + (ks & 1) * 2048;
            const i32x4 t0 = *(const i32x4*)(p + ra0), t1 = *(const i32x4*)(p + ra1);
            i32x8& d = xa[ks];
            d[0] = t0.x; d[1] = t0.y; d[2] = t0.z; d[3] = t0.w; d[4] = t1.x; d[5] = t1.y; d[6] = t1.z; d[7] = t1.w;
        };
        auto readA = [&](int half, int so) { readAq(2 * half, so); readAq(2 * half + 1, so); };
        const int q = i16 >> 1, pp = lane & 1;
        const int fB = ((q >> 2) & 1) | ((g & 1) << 1);
        int rb[4];
#pragma unroll
        for (int t = 0; t < 4; ++t) rb[t] = g * 1024 + q * 64 + ((t ^ fB) << 4) + pp * 8;
        i32x8 wb[4][4];
        auto readB = [&](int gI, int slot) {
            const int ks = gI >> 1, h = gI & 1;
#pragma unroll
            for (int t = 0; t < 4; ++t) {
#pragma unroll
                for (int r = 0; r < 2; ++r) {
                    const i32x2 v = __builtin_amdgcn_ds_read_tr8_b64_v2i32((LAS i32x2*)(lw + slot * 4096 + rb[t] + r * 512));
                    wb[ks][t][4 * h + 2 * r] = v.x; wb[ks][t][4 * h + 2 * r + 1] = v.y;
                }
            }
        };
        f32x4 acc[4];
        auto mm = [&](int ks) {
#pragma unroll
            for (int t = 0; t < 4; ++t) {
                const f32x4 c = ks == 0 ? f32x4{0.f, 0.f, 0.f, 0.f} : acc[t];
                acc[t] = __builtin_amdgcn_mfma_scale_f32_16x16x128_f8f6f4(wb[ks][t], xa[ks], c, 1, 0, 0, 0, 0, 0);
            }
        };
        auto mm2 = [&](int ks, int hf) {
#pragma unroll
            for (int t = 2 * hf; t < 2 * hf + 2; ++t) {
                const f32x4 c = ks == 0 ? f32x4{0.f, 0.f, 0.f, 0.f} : acc[t];
                acc[t] = __builtin_amdgcn_mfma_scale_f32_16x16x128_f8f6f4(wb[ks][t], xa[ks], c, 1, 0, 0, 0, 0, 0);
            }
        };
        const bool young = wv >= 4;
        unsigned char* const pw = lw + 4 * 4096 + i16 * 256;
        const int yrow = 4 * (wv & 3) + g;
        const unsigned char* const py = smem + 4 * 4096 + yrow * 256 + ((i16 ^ (yrow & 7)) << 4);
        char* optr = (char*)(late ? P.myout : P.out[lab]) + ((size_t)(orow0 + yrow) * NLOC + slice * 64 + i16 * 4) * 4;
        f32x4 ps[4], po;
        auto writeP = [&]() {
#pragma unroll
            for (int t = 0; t < 4; ++t) *(f32x4*)(pw + (((4 * t + g) ^ (i16 & 7)) << 4)) = acc[t];
        };
        auto readY = [&](int w0) {
#pragma unroll
            for (int j = 0; j < 4; ++j) ps[j] = *(const f32x4*)(py + (w0 + j) * WAVE_LDS);
        };
        auto emitY = [&]() {
            const f32x4 s = (po + ((ps[0] + ps[1]) + (ps[2] + ps[3]))) * scale;
            st16_sys(optr, s);
            optr += ostep;
        };

        WAIT_V(16); FINE_AT(3); readA(0, 0); WAIT_L0(); loadB(3, 0);
        WAIT_V(16); readA(1, 4096); WAIT_L0(); loadB(4, 1);
        asm volatile("" : "+s"(scx), "+s"(scw));
        scale = bad ? __builtin_nanf("") : scx * scw;
        WAIT_V(16); FINE_AT(4); readB(0, 2); WAIT_L0(); loadB(5, 2);
        WAIT_V(16); readB(1, 3); WAIT_L0(); loadB(6, 3); SCHED(); mm(0); SCHED();
        WAIT_V(16); readB(2, 4); WAIT_L0(); loadB(7, 4);
        SETPRIO(0);
        if (YBAR == 0 && young) BAR();
        WAIT_V(16); FINE_AT(6); readB(3, 0); SCHED(); mm(1); SCHED();
        WAIT_V(12); readB(4, 1);
        WAIT_V(8); readB(5, 2); WAIT_L0(); SCHED(); mm(2); SCHED();
        loadA2(1, 0, 4096);
        if (YBAR == 1 && young) BAR();
        WAIT_V(12); readB(6, 3);
        WAIT_V(8); FINE_AT(7); readB(7, 4); WAIT_L0(); SCHED(); mm(3); SCHED();
        loadA2(2, 2 * 4096, 3 * 4096);
        if (YBAR == 2 && young) BAR();
        readY(0);
        writeP();
        WAIT_V(8);
        readA(0, 0); readA(1, 4096);
        po = (ps[0] + ps[1]) + (ps[2] + ps[3]);
        WAIT_L0();
        FINE_AT(8);
        BAR();
        if (!aux) { if (remote) STAMP_AT(1); else STAMP_AT(2); } else if (!late) STAMP_AT(11);

        int sa = 0, sn = 2 * 4096;
        auto phaseX = [&](int i, bool yng) {
            const int nb = i + 2 < nblk ? i + 2 : nblk - 1;
            const uint8_t* p = aw + nb * astep;
            const unsigned l = lbase + sa;
#pragma unroll
            for (int ks = 0; ks < 4; ++ks) {
                SCHED();
                mm2(ks, 0);
                SCHED();
                glds16(p + ks * 1024, va0, l + ks * 2048);
                SCHED();
                mm2(ks, 1);
                SCHED();
                glds16(p + ks * 1024, va1, l + ks * 2048 + 1024);
                if (yng && ks == YRD) readY(4);
                CYC_AT(1 + ks);
            }
            SCHED();
        };
        if (!young) {
#pragma unroll 1
            for (int i = 1; i < nblk; ++i) {
                CYC_AT(0);
                if (i == 7) { CYC7(); }
                phaseX(i, false);
                CYC_AT(5);
                BAR();
                CYC_AT(6);
                WAIT_V(8);
                CYC_AT(7);
                readA(0, sn); readA(1, sn + 4096);
                CYC_AT(8);
                writeP();
                CYC_AT(9);
                { const int t = sa; sa = sn; sn = t; }
                WAIT_L0();
                CYC_AT(10);
                BAR();
            }
            BAR();
        } else {
#pragma unroll 1
            for (int i = 1; i < nblk; ++i) {
                CYC_AT(0);
                if (i == 7) { CYC7(); }
                phaseX(i, true);
                emitY();
                CYC_AT(5);
                BAR();
                CYC_AT(6);
                readY(0);
                if (__builtin_expect(i == 1, 0)) WAIT_V(9);
                WAIT_V(10);
                CYC_AT(7);
                readA(0, sn); readA(1, sn + 4096);
                CYC_AT(8);
                writeP();
                CYC_AT(9);
                { const int t = sa; sa = sn; sn = t; }
                po = (ps[0] + ps[1]) + (ps[2] + ps[3]);
                WAIT_L0();
                CYC_AT(10);
                BAR();
            }
            readY(4); emitY();
        }
        if (remote && !late) STAMP_AT(3);
        if (!aux) call = __builtin_amdgcn_readfirstlane(__hip_atomic_load(cnt, RLX_AGT)) + 1u;
    }
    WAIT_V(0);
    BAR();
    if (late) STAMP_AT(9); else if (remote && aux) STAMP_AT(12); else if (remote) STAMP_AT(4); else STAMP_AT(10);
    if (wv != 0) return;
    if (lane == 0) {
        if (dst != rank) __hip_atomic_store((unsigned*)(P.ws[lab] + WS_FLAG) + rank * 32 + ti, call, RLX_SYS);
        __hip_atomic_store(cnt, call, RLX_AGT);
    }
    if (!SOLO && !remote && ti == 31) {
        GAS char* fl = myws + WS_FLAG;
        const bool skip = (lane >> 3) == rank || (lane & 7) >= 4;
        const unsigned long long t0 = __builtin_amdgcn_s_memrealtime();
        for (;;) {
            u32x4 f;
            asm volatile("s_nop 4\n\tglobal_load_dwordx4 %0, %1, %2 sc0 sc1\n\ts_waitcnt vmcnt(0)" : "=v"(f) : "v"(lane * 16), "s"(fl) : "memory");
            const bool ok = skip || (f.x >= call && f.y >= call && f.z >= call && f.w >= call);
            if (__all(ok)) break;
            if (__builtin_amdgcn_s_memrealtime() - t0 > DEADLINE_TICKS) {
                if (lane == 0) {
                    __hip_atomic_store(tmo, call, RLX_AGT);
                    __hip_atomic_store(P.myout + (size_t)rank * MLOC * NLOC, __builtin_nanf(""), RLX_SYS);
                }
                break;
            }
            __builtin_amdgcn_s_sleep(1);
        }
        STAMP_AT(5);
    }
    STAMP_OUT();
}

extern "C" void kernel_launch(int rank, int nranks, ncclComm_t comm,
                              hipStream_t stream,
                              void* const* d_in, const int* in_sizes, int n_in,
                              void* d_out, int out_size,
                              const int* in_dtypes, int out_dtype,
                              void* const* d_out_all,
                              void* const* d_ws_all, size_t ws_bytes) {
    if (nranks != NR || (unsigned)rank >= (unsigned)NR || n_in < 4 || ws_bytes < WS_NEED ||
        in_sizes[0] != MLOC * KDIM || in_sizes[1] != KDIM * NDIM || out_size != NR * MLOC * NLOC) {
        fprintf(stderr, "kernel_launch: unexpected problem shape\n");
        return;
    }
    static bool attr_set = false;
    if (!attr_set) {
        HIP_CHECK(hipFuncSetAttribute((const void*)gemm_a2a, hipFuncAttributeMaxDynamicSharedMemorySize, LDS_BYTES));
        attr_set = true;
    }
    Params P;
    P.x = (const uint8_t*)d_in[0];
    P.w = (const uint8_t*)d_in[1];
    P.sx = (const float*)d_in[2];
    P.sw = (const float*)d_in[3];
    for (int p = 0; p < NR; ++p) { P.out[p] = (float*)d_out_all[p]; P.ws[p] = (char*)d_ws_all[p]; }
    P.myws = P.ws[rank];
    P.myout = P.out[rank];
    P.rank = rank;
    P.pad = 0;
    hipLaunchKernelGGL(gemm_a2a, dim3(NWG), dim3(512), LDS_BYTES, stream, P);
    HIP_CHECK(hipGetLastError());
}
```

```cpp
#include <hip/hip_runtime.h>
#include <rccl/rccl.h>
#include <cstdio>
#include <cstdint>

#define HIP_CHECK(expr) do { hipError_t _e = (expr); if (_e != hipSuccess) { \
    fprintf(stderr, "HIP error %s at %s:%d\n", hipGetErrorString(_e), __FILE__, __LINE__); \
    return; } } while (0)

#define XFB 6
#define NPUSH 4
#define PUSH_INFL (12 / NPUSH)

#define LATE_FENCE 0
#define PFORD 2
#define NOLATE 0

#define L1EARLY 1
#define NOSTORE 0
#define NOLOAD 0
#define XPRIO 1
#define PFGATE 550
#define PFINFL 12
#define PFINFL2 12
#define NG2 4
#define NL23 12
#define YRD 2
#define PUSH_WIN (16 / NPUSH)
#define YBAR 1
#define NOPUSH 0
#define PFSPLIT 1
#define NEARLY 5
#define PFGAP 100
#define A2LATE 1
#define TOPO 1
#define REVT 1
#define WPF 0
#define PRIO 0
#define SETPRIO(n) do {} while (0)

typedef int i32x8 __attribute__((ext_vector_type(8)));
typedef int i32x4 __attribute__((ext_vector_type(4)));
typedef int i32x2 __attribute__((ext_vector_type(2)));
typedef float f32x4 __attribute__((ext_vector_type(4)));
typedef unsigned u32x4 __attribute__((ext_vector_type(4)));
#define LAS __attribute__((address_space(3)))
#define GAS __attribute__((address_space(1)))
#define CAS __attribute__((address_space(4)))

constexpr int NR = 8;
constexpr int KDIM = 4096, NDIM = 4096, MLOC = 512, NLOC = 512;
constexpr int NBLK = MLOC / 16;
constexpr int SBLK = NBLK - XFB;
constexpr int SB0 = (SBLK + 1) / 2;
constexpr int WAVE_LDS = 20480;
constexpr int LDS_BYTES = 8 * WAVE_LDS;
constexpr int NWG = 256;
static_assert(XFB >= 2 && XFB <= 6 && NL23 >= 2 && NL23 % 2 == 0, "block split");

constexpr size_t WS_FLAG = 0;
constexpr size_t WS_XFLAG = 2048;
constexpr size_t WS_CNT = 4096;
constexpr size_t WS_TMO = 8192;
constexpr size_t WS_XBUF = 1u << 20;
constexpr size_t XSLOT = (size_t)XFB * 16 * KDIM;
constexpr size_t WS_NEED = WS_XBUF + 2 * NR * XSLOT;
constexpr unsigned long long DEADLINE_TICKS = 50000000ull;
#define STAMP_AT(k) do {} while (0)
#define FINE_AT(k) do {} while (0)
#define STAMP_OUT() do {} while (0)

#define CYC_AT(k) do {} while (0)
#define CYC7() do {} while (0)
#define SOLO 0

struct Params {
    const uint8_t* x;
    const uint8_t* w;
    char* myws;
    float* myout;
    const float* sx;
    const float* sw;
    int rank;
    unsigned lmap;
    float* out[NR];
    char* ws[NR];
};

__device__ __forceinline__ void st16_sys(void* p, f32x4 v) {
    asm volatile("global_store_dwordx4 %0, %1, off sc0 sc1\n\ts_nop 1" :: "v"(p), "v"(v) : "memory");
}
#define PAD ""
#define STR_(x) #x
#define STR(x) STR_(x)
#define WAIT_V(n) asm volatile("s_waitcnt vmcnt(" STR(n) ")" ::: "memory")
#define WAIT_VN(n) asm volatile("s_waitcnt vmcnt(%0)" :: "n"(n) : "memory")
#define WAIT_L0() do { __builtin_amdgcn_s_waitcnt(0xC07F); asm volatile("" ::: "memory"); } while (0)
#define BAR() asm volatile("s_barrier" ::: "memory")
#define SCHED() __builtin_amdgcn_sched_barrier(0)
#define RLX_SYS __ATOMIC_RELAXED, __HIP_MEMORY_SCOPE_SYSTEM
#define RLX_AGT __ATOMIC_RELAXED, __HIP_MEMORY_SCOPE_AGENT

__device__ __forceinline__ void glds16(const uint8_t* base, unsigned voff, unsigned lds) {
    asm volatile(PAD "s_mov_b32 m0, %2\n\ts_nop 0\n\tglobal_load_lds_dwordx4 %0, %1" :: "v"(voff), "s"(base), "s"(lds) : "memory");
}

__device__ __forceinline__ void glds16_sys(const uint8_t* base, unsigned voff, unsigned lds) {
    asm volatile(PAD "s_mov_b32 m0, %2\n\ts_nop 0\n\tglobal_load_lds_dwordx4 %0, %1 sc0 sc1" :: "v"(voff), "s"(base), "s"(lds) : "memory");
}

template <int I> struct IC { static constexpr int value = I; };
template <int N, int I = 0, class F> __device__ __forceinline__ void static_for(F&& f) {
    if constexpr (I < N) { f(IC<I>{}); static_for<N, I + 1>(f); }
}
constexpr int push_wait(int i, int n, int w, int f) {
    int c = 0;
    if (i < f) { c = (w - 1 - i) + i; for (int k = 0; k < i; ++k) c += (w + k < n); return c; }
    for (int k = i - f; k < i; ++k) c += (w + k < n) + (k > i - f);
    return c;
}

__global__ void __launch_bounds__(512, 2) gemm_a2a(Params P) {
    extern __shared__ __attribute__((aligned(16))) unsigned char smem[];
    const int lane = threadIdx.x & 63;
    const int wv = __builtin_amdgcn_readfirstlane(threadIdx.x >> 6);
    const int b = blockIdx.x;
    const int ti = REVT ? 31 - (b >> 3) : b >> 3;
    int rank = P.rank;
    GAS char* myws = (GAS char*)P.myws;
    const uint8_t* gx = P.x;
    const uint8_t* gw = P.w;
    unsigned lmap = P.lmap;
    asm volatile("" : "+s"(rank), "+s"(myws), "+s"(gx), "+s"(gw), "+s"(lmap));
    const int lab = (lmap >> ((b & 7) * 4)) & 7;
    GAS unsigned* const cnt = (GAS unsigned*)(myws + WS_CNT) + b;
    GAS unsigned* const tmo = (GAS unsigned*)(myws + WS_TMO);
    STAMP_AT(0);

    const int slice = ti & 7, grp = ti >> 3;
    const bool remote = lab != rank;
    const bool aux = ti >= 16;
    const bool late = remote && aux;
    const unsigned lbase = (unsigned)(uintptr_t)(LAS unsigned char*)smem + wv * WAVE_LDS;
    unsigned call = 0;

    const unsigned vp = (lane >> 5) * NDIM + (lane & 31) * 16;
    auto pfB = [&](const uint8_t* base, int m) {
        const int row = ((m >> 8) & 3) * 1024 + ((m >> 10) * 4 + ((m >> 5) & 3)) * 128 + ((m >> 7) & 1) * 64 + (m & 31) * 2;
        glds16(base + (size_t)row * NDIM, vp, lbase);
    };
    constexpr int WPFN = WPF * 64;
    if (WPF && !aux && wv < 4) {
#pragma unroll
        for (int j = 0; j < WPF; ++j) pfB(gw + lab * NLOC, j * 64 + ti * 4 + wv);
    }
    if (aux) {
        const int q = ti - 16;
        const unsigned long long tstart = __builtin_amdgcn_s_memrealtime();
        unsigned prev;
        if (q >= 16 - NPUSH) {
            asm volatile("s_nop 4\n\tglobal_load_dword %0, %1, %2 sc1" : "=v"(prev) : "v"(0u), "s"(cnt) : "memory");
            if (remote && !NOPUSH) {
                constexpr int NX = XFB * 8 / NPUSH;
                const unsigned xoff = ((q - (16 - NPUSH)) * 8 + wv) * (NX * 1024) + lane * 16;
                const uint8_t* src = gx + (size_t)SBLK * 16 * KDIM;
                constexpr int PW = NX < PUSH_WIN ? NX : PUSH_WIN;
                static_assert(PW == NX || PW > PUSH_INFL, "window");
                f32x4 v[NX];
#pragma unroll
                for (int i = 0; i < PW; ++i)
                    asm volatile(PAD "global_load_dwordx4 %0, %1, %2" : "=v"(v[i]) : "v"(xoff + i * 1024), "s"(src) : "memory");
                char* const pws = P.ws[lab];
                WAIT_VN(PW);
                asm volatile("" : "+v"(prev));
                call = __builtin_amdgcn_readfirstlane(prev) + 1u;
                char* d = pws + WS_XBUF + ((call & 1u) * NR + rank) * XSLOT + xoff;
                static_for<NX>([&](auto ic) {
                    constexpr int i = decltype(ic)::value;
                    WAIT_VN(push_wait(i, NX, PW, PUSH_INFL));
                    asm volatile("" : "+v"(v[i]));
                    st16_sys(d + i * 1024, v[i]);
                    if constexpr (PW + i < NX)
                        asm volatile(PAD "global_load_dwordx4 %0, %1, %2" : "=v"(v[PW + i]) : "v"(xoff + (PW + i) * 1024), "s"(src) : "memory");
                    if (i == 0) STAMP_AT(6);
                });
                WAIT_V(0);
                BAR();
                STAMP_AT(7);
                if (threadIdx.x == 0)
                    __hip_atomic_store((unsigned*)(pws + WS_XFLAG) + rank * 32 + (q - (16 - NPUSH)), call, RLX_SYS);
            }
            while (__builtin_amdgcn_s_memrealtime() - tstart < PFGATE) __builtin_amdgcn_s_sleep(2);
        } else {
            constexpr int NPW = (16 - NPUSH) * 8;
            const unsigned vl = lane * 16;
            const int u = q * 8 + wv;
            const uint8_t* wl = gw + lab * NLOC;
#pragma unroll 1
            for (int m = q * 4 + wv; wv < 4 && m < 128; m += NPW / 2) glds16(gx + m * 1024, vl, lbase);
#pragma unroll 1
            for (int m = q * 4 + (wv & 3); m < (2048 - WPFN) / 2; m += NPW / 2) { pfB(wl, WPFN + m + (wv >> 2) * ((2048 - WPFN) / 2)); WAIT_V(PFINFL); }
            asm volatile("s_nop 4\n\tglobal_load_dword %0, %1, %2 sc1" : "=v"(prev) : "v"(0u), "s"(cnt) : "memory");
            BAR();
            STAMP_AT(12);
#pragma unroll 1
            for (int m = 128 + u; m < NEARLY * 128; m += NPW) glds16(gx + m * 1024, vl, lbase);
            WAIT_V(0);
            STAMP_AT(13);
            {
                const unsigned long long t1 = __builtin_amdgcn_s_memrealtime();
                while (__builtin_amdgcn_s_memrealtime() - t1 < PFGAP) __builtin_amdgcn_s_sleep(2);
            }
            const int mend = (remote ? SBLK : NBLK) * 64;
#pragma unroll 1
            for (int m = NEARLY * 128 + u; m < mend; m += NPW) { glds16(gx + m * 1024, vl, lbase); WAIT_V(PFINFL2); }
            if (remote) {
                BAR();
                const uint8_t* wo = gw + rank * NLOC;
#pragma unroll 1
                for (int m = u; m < 2048; m += NPW) { pfB(wo, m); WAIT_V(PFINFL2); }
            }
        }

        WAIT_V(0);
        asm volatile("" : "+v"(prev));
        call = __builtin_amdgcn_readfirstlane(prev) + 1u;
    }

    const int dst = late ? rank : lab;
    int nblk, blk0;
    constexpr int bstr = 2;
    {
        const int nmain = remote ? SBLK : NBLK - NL23;
        if (late) { nblk = (XFB + 1 - (grp & 1)) >> 1; blk0 = grp & 1; }
        else if (!aux) { nblk = (nmain + 1 - grp) >> 1; blk0 = grp; }
        else { nblk = NL23 / 2; blk0 = nmain + (grp & 1); }
    }
    const uint8_t* abase = gx + (size_t)blk0 * 16 * KDIM;
    const int orow0 = late ? lab * MLOC + (SBLK + blk0) * 16 : rank * MLOC + blk0 * 16;
    constexpr int astep = bstr * 16 * KDIM, ostep = bstr * 16 * NLOC * 4;

    {
        bool bad = false;
        if (late) {
            if (wv == 0) {
                GAS unsigned* fl = (GAS unsigned*)(myws + WS_XFLAG) + lab * 32 + (lane & (NPUSH - 1));
                const unsigned long long t0 = __builtin_amdgcn_s_memrealtime();
                while (!SOLO && !NOPUSH && !__all(__hip_atomic_load(fl, RLX_SYS) >= call)) {
                    if (__hip_atomic_load(tmo, RLX_AGT) == call || __builtin_amdgcn_s_memrealtime() - t0 > DEADLINE_TICKS) {
                        __hip_atomic_store(tmo, call, RLX_AGT);
                        break;
                    }
                    __builtin_amdgcn_s_sleep(8);
                }
                WAIT_V(0);
            }
            BAR();
            bad = __builtin_amdgcn_readfirstlane(__hip_atomic_load(tmo, RLX_AGT)) == call;
            abase = (const uint8_t*)(char*)myws + WS_XBUF + ((call & 1u) * NR + lab) * XSLOT + (size_t)blk0 * 16 * KDIM;
            STAMP_AT(8);
        }

        unsigned char* const lw = smem + wv * WAVE_LDS;
        const int r8 = lane >> 3, c8 = lane & 7;
        const unsigned va0 = r8 * KDIM + ((c8 ^ (r8 >> 1)) << 4);
        const unsigned va1 = (8 + r8) * KDIM + ((c8 ^ (4 + (r8 >> 1))) << 4);
        const uint8_t* aw = abase + wv * 128;
        auto loadAq = [&](int blk, int ks, int so, bool sys) {
            const bool in = blk < (NOLOAD ? 1 : nblk);
            const uint8_t* p = aw + (in ? blk : 0) * astep + ks * 1024;
            const unsigned o0 = in ? va0 : 0u, o1 = in ? va1 : 0u;
            const unsigned l = lbase + so + (ks & 1) * 2048;
            if (sys) { glds16_sys(p, o0, l); glds16_sys(p, o1, l + 1024); }
            else { glds16(p, o0, l); glds16(p, o1, l + 1024); }
        };
        auto loadA2 = [&](int blk, int so0, int so1) {
            if (late) { loadAq(blk, 0, so0, true); loadAq(blk, 1, so0, true); loadAq(blk, 2, so1, true); loadAq(blk, 3, so1, true); }
            else { loadAq(blk, 0, so0, false); loadAq(blk, 1, so0, false); loadAq(blk, 2, so1, false); loadAq(blk, 3, so1, false); }
        };
        const int r16 = lane >> 2, c4 = lane & 3;
        const unsigned vb0 = r16 * NDIM + ((c4 ^ ((r16 >> 2) & 1)) << 4);
        const unsigned vb1 = (16 + r16) * NDIM + ((c4 ^ (((r16 >> 2) & 1) | 2)) << 4);
        const uint8_t* bw = gw + (size_t)wv * 128 * NDIM + dst * NLOC + slice * 64;
        auto loadB = [&](int gI, int slot) {
            const uint8_t* p = bw + (size_t)((gI >> 1) * 1024 + (gI & 1) * 64) * NDIM;
            const unsigned l = lbase + slot * 4096;
            glds16(p, vb0, l); glds16(p, vb1, l + 1024);
            glds16(p + 32 * NDIM, vb0, l + 2048); glds16(p + 32 * NDIM, vb1, l + 3072);
        };
        FINE_AT(1);
        SETPRIO(1);
        loadA2(0, 0, 4096);
        loadB(0, 2); loadB(1, 3); loadB(2, 4);
        FINE_AT(2);

        float scx, scw, scale;
        asm volatile(PAD "s_load_dword %0, %2, 0x0\n\ts_load_dword %1, %3, 0x0" : "=&s"(scx), "=&s"(scw) : "s"(P.sx), "s"(P.sw) : "memory");
        const int i16 = lane & 15, g = lane >> 4;
        const int ra0 = i16 * 128 + ((g ^ (i16 >> 1)) << 4);
        const int ra1 = i16 * 128 + (((4 + g) ^ (i16 >> 1)) << 4);
        i32x8 xa[4];
        auto readAq = [&](int ks, int so) {
            const unsigned char* p = lw + so + (ks & 1) * 2048;
            const i32x4 t0 = *(const i32x4*)(p + ra0), t1 = *(const i32x4*)(p + ra1);
            i32x8& d = xa[ks];
            d[0] = t0.x; d[1] = t0.y; d[2] = t0.z; d[3] = t0.w; d[4] = t1.x; d[5] = t1.y; d[6] = t1.z; d[7] = t1.w;
        };
        auto readA = [&](int half, int so) { readAq(2 * half, so); readAq(2 * half + 1, so); };
        const int q = i16 >> 1, pp = lane & 1;
        const int fB = ((q >> 2) & 1) | ((g & 1) << 1);
        int rb[4];
#pragma unroll
        for (int t = 0; t < 4; ++t) rb[t] = g * 1024 + q * 64 + ((t ^ fB) << 4) + pp * 8;
        i32x8 wb[4][4];
        auto readB = [&](int gI, int slot) {
            const int ks = gI >> 1, h = gI & 1;
#pragma unroll
            for (int t = 0; t < 4; ++t) {
#pragma unroll
                for (int r = 0; r < 2; ++r) {
                    const i32x2 v = __builtin_amdgcn_ds_read_tr8_b64_v2i32((LAS i32x2*)(lw + slot * 4096 + rb[t] + r * 512));
                    wb[ks][t][4 * h + 2 * r] = v.x; wb[ks][t][4 * h + 2 * r + 1] = v.y;
                }
            }
        };
        f32x4 acc[4];
        auto mm = [&](int ks) {
#pragma unroll
            for (int t = 0; t < 4; ++t) {
                const f32x4 c = ks == 0 ? f32x4{0.f, 0.f, 0.f, 0.f} : acc[t];
                acc[t] = __builtin_amdgcn_mfma_scale_f32_16x16x128_f8f6f4(wb[ks][t], xa[ks], c, 1, 0, 0, 0, 0, 0);
            }
        };
        auto mm2 = [&](int ks, int hf) {
#pragma unroll
            for (int t = 2 * hf; t < 2 * hf + 2; ++t) {
                const f32x4 c = ks == 0 ? f32x4{0.f, 0.f, 0.f, 0.f} : acc[t];
                acc[t] = __builtin_amdgcn_mfma_scale_f32_16x16x128_f8f6f4(wb[ks][t], xa[ks], c, 1, 0, 0, 0, 0, 0);
            }
        };
        const bool young = wv >= 4;
        unsigned char* const pw = lw + 4 * 4096 + i16 * 256;
        const int yrow = 4 * (wv & 3) + g;
        const unsigned char* const py = smem + 4 * 4096 + yrow * 256 + ((i16 ^ (yrow & 7)) << 4);
        char* optr = (char*)(late ? P.myout : P.out[lab]) + ((size_t)(orow0 + yrow) * NLOC + slice * 64 + i16 * 4) * 4;
        f32x4 ps[4], po;
        auto writeP = [&]() {
#pragma unroll
            for (int t = 0; t < 4; ++t) *(f32x4*)(pw + (((4 * t + g) ^ (i16 & 7)) << 4)) = acc[t];
        };
        auto readY = [&](int w0) {
#pragma unroll
            for (int j = 0; j < 4; ++j) ps[j] = *(const f32x4*)(py + (w0 + j) * WAVE_LDS);
        };
        auto emitY = [&]() {
            const f32x4 s = (po + ((ps[0] + ps[1]) + (ps[2] + ps[3]))) * scale;
            st16_sys(optr, s);
            optr += ostep;
        };

        WAIT_V(16); FINE_AT(3); readA(0, 0); WAIT_L0(); loadB(3, 0);
        WAIT_V(16); readA(1, 4096); WAIT_L0(); loadB(4, 1);
        asm volatile("" : "+s"(scx), "+s"(scw));
        scale = bad ? __builtin_nanf("") : scx * scw;
        WAIT_V(16); FINE_AT(4); readB(0, 2); WAIT_L0(); loadB(5, 2);
        WAIT_V(16); readB(1, 3); WAIT_L0(); loadB(6, 3); SCHED(); mm(0); SCHED();
        WAIT_V(16); readB(2, 4); WAIT_L0(); loadB(7, 4);
        SETPRIO(0);
        if (young) BAR();
        WAIT_V(16); FINE_AT(6); readB(3, 0); SCHED(); mm(1); SCHED();
        WAIT_V(12); readB(4, 1);
        WAIT_V(8); readB(5, 2); WAIT_L0(); SCHED(); mm(2); SCHED();
        if (young) { loadA2(1, 0, 4096); WAIT_V(12); } else WAIT_V(4);
        readB(6, 3);
        if (young) WAIT_V(8); else WAIT_V(0);
        FINE_AT(7); readB(7, 4); WAIT_L0(); SCHED(); mm(3); SCHED();
        readY(0);
        writeP();
        if (!young) { WAIT_L0(); BAR(); loadA2(1, 0, 4096); WAIT_V(0); }
        loadA2(2, 2 * 4096, 3 * 4096);
        WAIT_V(8);
        readA(0, 0); readA(1, 4096);
        po = (ps[0] + ps[1]) + (ps[2] + ps[3]);
        WAIT_L0();
        FINE_AT(8);
        if (young) BAR();
        if (!aux) { if (remote) STAMP_AT(1); else STAMP_AT(2); } else if (!late) STAMP_AT(11);

        int sa = 0, sn = 2 * 4096;
        auto phaseX = [&](int i, bool yng) {
            const int nb = i + 2 < nblk ? i + 2 : nblk - 1;
            const uint8_t* p = aw + nb * astep;
            const unsigned l = lbase + sa;
#pragma unroll
            for (int ks = 0; ks < 4; ++ks) {
                SCHED();
                mm2(ks, 0);
                SCHED();
                glds16(p + ks * 1024, va0, l + ks * 2048);
                SCHED();
                mm2(ks, 1);
                SCHED();
                glds16(p + ks * 1024, va1, l + ks * 2048 + 1024);
                if (yng && ks == YRD) readY(4);
                CYC_AT(1 + ks);
            }
            SCHED();
        };
        if (!young) {
#pragma unroll 1
            for (int i = 1; i < nblk; ++i) {
                CYC_AT(0);
                if (i == 7) { CYC7(); }
                phaseX(i, false);
                CYC_AT(5);
                BAR();
                CYC_AT(6);
                WAIT_V(8);
                CYC_AT(7);
                readA(0, sn); readA(1, sn + 4096);
                CYC_AT(8);
                writeP();
                CYC_AT(9);
                { const int t = sa; sa = sn; sn = t; }
                WAIT_L0();
                CYC_AT(10);
                BAR();
            }
            BAR();
        } else {
#pragma unroll 1
            for (int i = 1; i < nblk; ++i) {
                CYC_AT(0);
                if (i == 7) { CYC7(); }
                phaseX(i, true);
                emitY();
                CYC_AT(5);
                BAR();
                CYC_AT(6);
                readY(0);
                if (__builtin_expect(i == 1, 0)) WAIT_V(9);
                WAIT_V(10);
                CYC_AT(7);
                readA(0, sn); readA(1, sn + 4096);
                CYC_AT(8);
                writeP();
                CYC_AT(9);
                { const int t = sa; sa = sn; sn = t; }
                po = (ps[0] + ps[1]) + (ps[2] + ps[3]);
                WAIT_L0();
                CYC_AT(10);
                BAR();
            }
            readY(4); emitY();
        }
        if (remote && !late) STAMP_AT(3);
        if (!aux) call = __builtin_amdgcn_readfirstlane(__hip_atomic_load(cnt, RLX_AGT)) + 1u;
    }
    WAIT_V(0);
    BAR();
    if (late) STAMP_AT(9); else if (remote) STAMP_AT(4); else STAMP_AT(10);
    if (wv != 0) return;
    if (lane == 0) {
        if (dst != rank) __hip_atomic_store((unsigned*)(P.ws[lab] + WS_FLAG) + rank * 32 + ti, call, RLX_SYS);
        __hip_atomic_store(cnt, call, RLX_AGT);
    }
    if (!SOLO && !remote && ti == 31) {
        GAS char* fl = myws + WS_FLAG;
        const bool skip = (lane >> 3) == rank || (lane & 7) >= 4;
        const unsigned long long t0 = __builtin_amdgcn_s_memrealtime();
        for (;;) {
            u32x4 f;
            asm volatile("s_nop 4\n\tglobal_load_dwordx4 %0, %1, %2 sc0 sc1\n\ts_waitcnt vmcnt(0)" : "=v"(f) : "v"(lane * 16), "s"(fl) : "memory");
            const bool ok = skip || (f.x >= call && f.y >= call && f.z >= call && f.w >= call);
            if (__all(ok)) break;
            if (__builtin_amdgcn_s_memrealtime() - t0 > DEADLINE_TICKS) {
                if (lane == 0) {
                    __hip_atomic_store(tmo, call, RLX_AGT);
                    __hip_atomic_store(P.myout + (size_t)rank * MLOC * NLOC, __builtin_nanf(""), RLX_SYS);
                }
                break;
            }
            __builtin_amdgcn_s_sleep(1);
        }
        STAMP_AT(5);
    }
    STAMP_OUT();
}

extern "C" void kernel_launch(int rank, int nranks, ncclComm_t comm,
                              hipStream_t stream,
                              void* const* d_in, const int* in_sizes, int n_in,
                              void* d_out, int out_size,
                              const int* in_dtypes, int out_dtype,
                              void* const* d_out_all,
                              void* const* d_ws_all, size_t ws_bytes) {
    if (nranks != NR || (unsigned)rank >= (unsigned)NR || n_in < 4 || ws_bytes < WS_NEED ||
        in_sizes[0] != MLOC * KDIM || in_sizes[1] != KDIM * NDIM || out_size != NR * MLOC * NLOC) {
        fprintf(stderr, "kernel_launch: unexpected problem shape\n");
        return;
    }
    static bool attr_set = false;
    if (!attr_set) {
        HIP_CHECK(hipFuncSetAttribute((const void*)gemm_a2a, hipFuncAttributeMaxDynamicSharedMemorySize, LDS_BYTES));
        attr_set = true;
    }
    Params P;
    P.x = (const uint8_t*)d_in[0];
    P.w = (const uint8_t*)d_in[1];
    P.sx = (const float*)d_in[2];
    P.sw = (const float*)d_in[3];
    for (int p = 0; p < NR; ++p) { P.out[p] = (float*)d_out_all[p]; P.ws[p] = (char*)d_ws_all[p]; }
    P.myws = P.ws[rank];
    P.myout = P.out[rank];
    P.rank = rank;
    {
        const int half[2][4] = {{0, 2, 3, 6}, {1, 4, 5, 7}};
        const int h = (rank == 1 || rank == 4 || rank == 5 || rank == 7);
        P.lmap = 0;
        for (int j = 0; j < 8; ++j) P.lmap |= (unsigned)(TOPO ? half[j < 4 ? h : 1 - h][j & 3] : j) << (4 * j);
    }
    hipLaunchKernelGGL(gemm_a2a, dim3(NWG), dim3(512), LDS_BYTES, stream, P);
    HIP_CHECK(hipGetLastError());
}
```

```cpp
#include <hip/hip_runtime.h>
#include <rccl/rccl.h>
#include <cstdio>
#include <cstdint>

#define HIP_CHECK(expr) do { hipError_t _e = (expr); if (_e != hipSuccess) { \
    fprintf(stderr, "HIP error %s at %s:%d\n", hipGetErrorString(_e), __FILE__, __LINE__); \
    return; } } while (0)

#define XFB 6
#define NL23 12
#define NEARLY 5
#define PFGAP 100
#define PFGATE 550
#define TOPO 1
#define SOLO 0

typedef int i32x8 __attribute__((ext_vector_type(8)));
typedef int i32x4 __attribute__((ext_vector_type(4)));
typedef int i32x2 __attribute__((ext_vector_type(2)));
typedef float f32x4 __attribute__((ext_vector_type(4)));
typedef unsigned u32x4 __attribute__((ext_vector_type(4)));
#define LAS __attribute__((address_space(3)))
#define GAS __attribute__((address_space(1)))

constexpr int NR = 8;
constexpr int KDIM = 4096, NDIM = 4096, MLOC = 512, NLOC = 512;
constexpr int NBLK = MLOC / 16;
constexpr int SBLK = NBLK - XFB;
constexpr int NPUSH = 4;
constexpr int PUSH_WIN = 4, PUSH_INFL = 3;
constexpr int WAVE_LDS = 20480;
constexpr int LDS_BYTES = 8 * WAVE_LDS;
constexpr int NWG = 256;
static_assert(XFB >= 2 && XFB <= 8 && NL23 >= 2 && NL23 % 2 == 0 && 2 * NEARLY <= SBLK, "block split");

constexpr size_t WS_FLAG = 0;
constexpr size_t WS_XFLAG = 2048;
constexpr size_t WS_CNT = 4096;
constexpr size_t WS_XBUF = 1u << 20;
constexpr size_t XSLOT = (size_t)XFB * 16 * KDIM;
constexpr size_t WS_NEED = WS_XBUF + 2 * NR * XSLOT;
constexpr unsigned long long DEADLINE_TICKS = 50000000ull;

#define STAMP_AT(k) do {} while (0)
#define STAMP_OUT() do {} while (0)

struct Params {
    const uint8_t* x;
    const uint8_t* w;
    char* myws;
    float* myout;
    const float* sx;
    const float* sw;
    int rank;
    unsigned lmap;
    float* out[NR];
    char* ws[NR];
};

#define STR_(x) #x
#define STR(x) STR_(x)
#define WAIT_V(n) asm volatile("s_waitcnt vmcnt(" STR(n) ")" ::: "memory")
#define WAIT_VN(n) asm volatile("s_waitcnt vmcnt(%0)" :: "n"(n) : "memory")
#define WAIT_L0() do { __builtin_amdgcn_s_waitcnt(0xC07F); asm volatile("" ::: "memory"); } while (0)
#define BAR() asm volatile("s_barrier" ::: "memory")
#define SCHED() __builtin_amdgcn_sched_barrier(0)
#define RLX_SYS __ATOMIC_RELAXED, __HIP_MEMORY_SCOPE_SYSTEM
#define RLX_AGT __ATOMIC_RELAXED, __HIP_MEMORY_SCOPE_AGENT

__device__ __forceinline__ void st16_sys(void* p, f32x4 v) {
    asm volatile("global_store_dwordx4 %0, %1, off sc0 sc1\n\ts_nop 1" :: "v"(p), "v"(v) : "memory");
}
__device__ __forceinline__ void glds16(const uint8_t* base, unsigned voff, unsigned lds) {
    asm volatile("s_mov_b32 m0, %2\n\ts_nop 0\n\tglobal_load_lds_dwordx4 %0, %1" :: "v"(voff), "s"(base), "s"(lds) : "memory");
}
__device__ __forceinline__ void glds16_sys(const uint8_t* base, unsigned voff, unsigned lds) {
    asm volatile("s_mov_b32 m0, %2\n\ts_nop 0\n\tglobal_load_lds_dwordx4 %0, %1 sc0 sc1" :: "v"(voff), "s"(base), "s"(lds) : "memory");
}

template <int I> struct IC { static constexpr int value = I; };
template <int N, int I = 0, class F> __device__ __forceinline__ void static_for(F&& f) {
    if constexpr (I < N) { f(IC<I>{}); static_for<N, I + 1>(f); }
}
constexpr int push_wait(int i, int n, int w, int f) {
    int c = 0;
    if (i < f) { c = w - 1; for (int k = 0; k < i; ++k) c += (w + k < n); return c; }
    for (int k = i - f; k < i; ++k) c += (w + k < n) + (k > i - f);
    return c;
}

__global__ void __launch_bounds__(512, 2) gemm_a2a(Params P) {
    extern __shared__ __attribute__((aligned(16))) unsigned char smem[];
    const int lane = threadIdx.x & 63;
    const int wv = __builtin_amdgcn_readfirstlane(threadIdx.x >> 6);
    const int b = blockIdx.x;
    const int ti = 31 - (b >> 3);
    int rank = P.rank;
    GAS char* myws = (GAS char*)P.myws;
    const uint8_t* gx = P.x;
    const uint8_t* gw = P.w;
    unsigned lmap = P.lmap;
    asm volatile("" : "+s"(rank), "+s"(myws), "+s"(gx), "+s"(gw), "+s"(lmap));
    const int lab = (lmap >> ((b & 7) * 4)) & 7;
    GAS unsigned* const cnt = (GAS unsigned*)(myws + WS_CNT) + b;
    GAS unsigned* const tmo = (GAS unsigned*)(myws + WS_FLAG) + rank * 32;
    STAMP_AT(0);

    const int slice = ti & 7, grp = ti >> 3;
    const bool remote = lab != rank;
    const bool aux = ti >= 16;
    const bool late = remote && aux;
    const unsigned lbase = (unsigned)(uintptr_t)(LAS unsigned char*)smem + wv * WAVE_LDS;
    unsigned call = 0;

    if (aux) {
        const int q = ti - 16;
        unsigned prev;
        if (q >= 16 - NPUSH) {
            const unsigned long long tstart = __builtin_amdgcn_s_memrealtime();
            asm volatile("s_nop 4\n\tglobal_load_dword %0, %1, %2 sc1" : "=v"(prev) : "v"(0u), "s"(cnt) : "memory");
            if (remote) {
                constexpr int NX = XFB * 8 / NPUSH;
                const unsigned xoff = ((q - (16 - NPUSH)) * 8 + wv) * (NX * 1024) + lane * 16;
                const uint8_t* src = gx + (size_t)SBLK * 16 * KDIM;
                f32x4 v[NX];
#pragma unroll
                for (int i = 0; i < PUSH_WIN; ++i)
                    asm volatile("global_load_dwordx4 %0, %1, %2" : "=v"(v[i]) : "v"(xoff + i * 1024), "s"(src) : "memory");
                char* const pws = P.ws[lab];
                WAIT_VN(PUSH_WIN);
                asm volatile("" : "+v"(prev));
                call = __builtin_amdgcn_readfirstlane(prev) + 1u;
                char* d = pws + WS_XBUF + ((call & 1u) * NR + rank) * XSLOT + xoff;
                static_for<NX>([&](auto ic) {
                    constexpr int i = decltype(ic)::value;
                    WAIT_VN(push_wait(i, NX, PUSH_WIN, PUSH_INFL));
                    asm volatile("" : "+v"(v[i]));
                    st16_sys(d + i * 1024, v[i]);
                    if constexpr (PUSH_WIN + i < NX)
                        asm volatile("global_load_dwordx4 %0, %1, %2" : "=v"(v[PUSH_WIN + i]) : "v"(xoff + (PUSH_WIN + i) * 1024), "s"(src) : "memory");
                    if (i == 0) STAMP_AT(6);
                });
                WAIT_V(0);
                BAR();
                STAMP_AT(7);
                if (threadIdx.x == 0)
                    __hip_atomic_store((unsigned*)(pws + WS_XFLAG) + rank * 32 + (q - (16 - NPUSH)), call, RLX_SYS);
            }
            while (__builtin_amdgcn_s_memrealtime() - tstart < PFGATE) __builtin_amdgcn_s_sleep(2);
        } else {
            constexpr int NPW = (16 - NPUSH) * 8;
            const unsigned vl = lane * 16;
            const unsigned vp = (lane >> 5) * NDIM + (lane & 31) * 16;
            const int u = q * 8 + wv;
            auto pfB = [&](const uint8_t* base, int m) {
                const int row = ((m >> 8) & 3) * 1024 + ((m >> 10) * 4 + ((m >> 5) & 3)) * 128 + ((m >> 7) & 1) * 64 + (m & 31) * 2;
                glds16(base + (size_t)row * NDIM, vp, lbase);
            };
#pragma unroll 1
            for (int m = q * 4 + wv; wv < 4 && m < 128; m += NPW / 2) glds16(gx + m * 1024, vl, lbase);
            const uint8_t* wl = gw + lab * NLOC;
#pragma unroll 1
            for (int m = q * 4 + (wv & 3); m < 1024; m += NPW / 2) { pfB(wl, m + (wv >> 2) * 1024); WAIT_V(12); }
            asm volatile("s_nop 4\n\tglobal_load_dword %0, %1, %2 sc1" : "=v"(prev) : "v"(0u), "s"(cnt) : "memory");
            BAR();
            STAMP_AT(12);
#pragma unroll 1
            for (int m = 128 + u; m < NEARLY * 128; m += NPW) glds16(gx + m * 1024, vl, lbase);
            WAIT_V(0);
            STAMP_AT(13);
            const unsigned long long t1 = __builtin_amdgcn_s_memrealtime();
            while (__builtin_amdgcn_s_memrealtime() - t1 < PFGAP) __builtin_amdgcn_s_sleep(2);
            const int mend = (remote ? SBLK : NBLK) * 64;
#pragma unroll 1
            for (int m = NEARLY * 128 + u; m < mend; m += NPW) { glds16(gx + m * 1024, vl, lbase); WAIT_V(12); }
            if (remote) {
                BAR();
                const uint8_t* wo = gw + rank * NLOC;
#pragma unroll 1
                for (int m = u; m < 2048; m += NPW) { pfB(wo, m); WAIT_V(12); }
            }
        }
        WAIT_V(0);
        asm volatile("" : "+v"(prev));
        call = __builtin_amdgcn_readfirstlane(prev) + 1u;
    }

    const int dst = late ? rank : lab;
    int nblk, blk0;
    {
        const int nmain = remote ? SBLK : NBLK - NL23;
        if (late) { nblk = (XFB + 1 - (grp & 1)) >> 1; blk0 = grp & 1; }
        else if (!aux) { nblk = (nmain + 1 - grp) >> 1; blk0 = grp; }
        else { nblk = NL23 / 2; blk0 = nmain + (grp & 1); }
    }
    const uint8_t* abase = gx + (size_t)blk0 * 16 * KDIM;
    const int orow0 = late ? lab * MLOC + (SBLK + blk0) * 16 : rank * MLOC + blk0 * 16;
    constexpr int astep = 2 * 16 * KDIM, ostep = 2 * 16 * NLOC * 4;
    bool bad = false;
    if (late) {
        if (wv == 0) {
            GAS unsigned* fl = (GAS unsigned*)(myws + WS_XFLAG) + lab * 32 + (lane & (NPUSH - 1));
            const unsigned long long t0 = __builtin_amdgcn_s_memrealtime();
            while (!SOLO && !__all(__hip_atomic_load(fl, RLX_SYS) >= call)) {
                if (__hip_atomic_load(tmo, RLX_AGT) == call || __builtin_amdgcn_s_memrealtime() - t0 > DEADLINE_TICKS) {
                    __hip_atomic_store(tmo, call, RLX_AGT);
                    break;
                }
                __builtin_amdgcn_s_sleep(8);
            }
            WAIT_V(0);
        }
        BAR();
        bad = __builtin_amdgcn_readfirstlane(__hip_atomic_load(tmo, RLX_AGT)) == call;
        if (!bad) abase = (const uint8_t*)(char*)myws + WS_XBUF + ((call & 1u) * NR + lab) * XSLOT + (size_t)blk0 * 16 * KDIM;
        STAMP_AT(8);
    }

    unsigned char* const lw = smem + wv * WAVE_LDS;
    const int r8 = lane >> 3, c8 = lane & 7;
    const unsigned va0 = r8 * KDIM + ((c8 ^ (r8 >> 1)) << 4);
    const unsigned va1 = (8 + r8) * KDIM + ((c8 ^ (4 + (r8 >> 1))) << 4);
    const uint8_t* aw = abase + wv * 128;
    auto loadA = [&](int blk, int so) {
        const uint8_t* p = aw + (blk < nblk ? blk : nblk - 1) * astep;
#pragma unroll
        for (int ks = 0; ks < 4; ++ks) {
            glds16_sys(p + ks * 1024, va0, lbase + so + ks * 2048);
            glds16_sys(p + ks * 1024, va1, lbase + so + ks * 2048 + 1024);
        }
    };
    const int r16 = lane >> 2, c4 = lane & 3;
    const unsigned vb0 = r16 * NDIM + ((c4 ^ ((r16 >> 2) & 1)) << 4);
    const unsigned vb1 = (16 + r16) * NDIM + ((c4 ^ (((r16 >> 2) & 1) | 2)) << 4);
    const uint8_t* bw = gw + (size_t)wv * 128 * NDIM + dst * NLOC + slice * 64;
    auto loadB = [&](int gI, int slot) {
        const uint8_t* p = bw + (size_t)((gI >> 1) * 1024 + (gI & 1) * 64) * NDIM;
        const unsigned l = lbase + slot * 4096;
        glds16(p, vb0, l); glds16(p, vb1, l + 1024);
        glds16(p + 32 * NDIM, vb0, l + 2048); glds16(p + 32 * NDIM, vb1, l + 3072);
    };
    loadA(0, 0);
    loadB(0, 2); loadB(1, 3); loadB(2, 4);

    float scx, scw, scale;
    asm volatile("s_load_dword %0, %2, 0x0\n\ts_load_dword %1, %3, 0x0" : "=&s"(scx), "=&s"(scw) : "s"(P.sx), "s"(P.sw) : "memory");
    const int i16 = lane & 15, g = lane >> 4;
    const int ra0 = i16 * 128 + ((g ^ (i16 >> 1)) << 4);
    const int ra1 = i16 * 128 + (((4 + g) ^ (i16 >> 1)) << 4);
    i32x8 xa[4];
    auto readA = [&](int k0, int k1, int so) {
#pragma unroll
        for (int ks = k0; ks < k1; ++ks) {
            const unsigned char* p = lw + so + ks * 2048;
            const i32x4 t0 = *(const i32x4*)(p + ra0), t1 = *(const i32x4*)(p + ra1);
            i32x8& d = xa[ks];
            d[0] = t0.x; d[1] = t0.y; d[2] = t0.z; d[3] = t0.w; d[4] = t1.x; d[5] = t1.y; d[6] = t1.z; d[7] = t1.w;
        }
    };
    const int q8 = i16 >> 1, pp = lane & 1;
    const int fB = ((q8 >> 2) & 1) | ((g & 1) << 1);
    int rb[4];
#pragma unroll
    for (int t = 0; t < 4; ++t) rb[t] = g * 1024 + q8 * 64 + ((t ^ fB) << 4) + pp * 8;
    i32x8 wb[4][4];
    auto readB = [&](int gI, int slot) {
        const int ks = gI >> 1, h = gI & 1;
#pragma unroll
        for (int t = 0; t < 4; ++t) {
#pragma unroll
            for (int r = 0; r < 2; ++r) {
                const i32x2 v = __builtin_amdgcn_ds_read_tr8_b64_v2i32((LAS i32x2*)(lw + slot * 4096 + rb[t] + r * 512));
                wb[ks][t][4 * h + 2 * r] = v.x; wb[ks][t][4 * h + 2 * r + 1] = v.y;
            }
        }
    };
    f32x4 acc[4];
    auto mm = [&](int ks, int t0, int t1) {
#pragma unroll
        for (int t = t0; t < t1; ++t) {
            const f32x4 c = ks == 0 ? f32x4{0.f, 0.f, 0.f, 0.f} : acc[t];
            acc[t] = __builtin_amdgcn_mfma_scale_f32_16x16x128_f8f6f4(wb[ks][t], xa[ks], c, 1, 0, 0, 0, 0, 0);
        }
    };
    const bool young = wv >= 4;
    unsigned char* const pw = lw + 4 * 4096 + i16 * 256;
    const int yrow = 4 * (wv & 3) + g;
    const unsigned char* const py = smem + 4 * 4096 + yrow * 256 + ((i16 ^ (yrow & 7)) << 4);
    char* optr = (char*)(late ? P.myout : P.out[lab]) + ((size_t)(orow0 + yrow) * NLOC + slice * 64 + i16 * 4) * 4;
    f32x4 ps[4], po;
    auto writeP = [&]() {
#pragma unroll
        for (int t = 0; t < 4; ++t) *(f32x4*)(pw + (((4 * t + g) ^ (i16 & 7)) << 4)) = acc[t];
    };
    auto readY = [&](int w0) {
#pragma unroll
        for (int j = 0; j < 4; ++j) ps[j] = *(const f32x4*)(py + (w0 + j) * WAVE_LDS);
    };
    auto emitY = [&]() {
        const f32x4 s = (po + ((ps[0] + ps[1]) + (ps[2] + ps[3]))) * scale;
        st16_sys(optr, s);
        optr += ostep;
    };

    WAIT_V(16); readA(0, 2, 0); WAIT_L0(); loadB(3, 0);
    WAIT_V(16); readA(2, 4, 0); WAIT_L0(); loadB(4, 1);
    asm volatile("" : "+s"(scx), "+s"(scw));
    scale = bad ? __builtin_nanf("") : scx * scw;
    WAIT_V(16); readB(0, 2); WAIT_L0(); loadB(5, 2);
    WAIT_V(16); readB(1, 3); WAIT_L0(); loadB(6, 3); SCHED(); mm(0, 0, 4); SCHED();
    WAIT_V(16); readB(2, 4); WAIT_L0(); loadB(7, 4);
    if (young) BAR();
    WAIT_V(16); readB(3, 0); SCHED(); mm(1, 0, 4); SCHED();
    WAIT_V(12); readB(4, 1);
    WAIT_V(8); readB(5, 2); WAIT_L0(); SCHED(); mm(2, 0, 4); SCHED();
    if (young) { loadA(1, 0); WAIT_V(12); } else WAIT_V(4);
    readB(6, 3);
    if (young) WAIT_V(8); else WAIT_V(0);
    readB(7, 4); WAIT_L0(); SCHED(); mm(3, 0, 4); SCHED();
    readY(0);
    writeP();
    if (!young) { WAIT_L0(); BAR(); loadA(1, 0); WAIT_V(0); }
    loadA(2, 2 * 4096);
    WAIT_V(8);
    readA(0, 4, 0);
    po = (ps[0] + ps[1]) + (ps[2] + ps[3]);
    WAIT_L0();
    if (young) BAR();
    if (!aux) { if (remote) STAMP_AT(1); else STAMP_AT(2); } else if (!late) STAMP_AT(11);

    int sa = 0, sn = 2 * 4096;
    auto phaseX = [&](int i, bool yng) {
        const uint8_t* p = aw + (i + 2 < nblk ? i + 2 : nblk - 1) * astep;
        const unsigned l = lbase + sa;
#pragma unroll
        for (int ks = 0; ks < 4; ++ks) {
            SCHED();
            mm(ks, 0, 2);
            SCHED();
            glds16_sys(p + ks * 1024, va0, l + ks * 2048);
            SCHED();
            mm(ks, 2, 4);
            SCHED();
            glds16_sys(p + ks * 1024, va1, l + ks * 2048 + 1024);
            if (yng && ks == 2) readY(4);
        }
        SCHED();
    };
    if (!young) {
#pragma unroll 1
        for (int i = 1; i < nblk; ++i) {
            phaseX(i, false);
            BAR();
            WAIT_V(8);
            readA(0, 4, sn);
            writeP();
            { const int t = sa; sa = sn; sn = t; }
            WAIT_L0();
            BAR();
        }
        BAR();
    } else {
#pragma unroll 1
        for (int i = 1; i < nblk; ++i) {
            phaseX(i, true);
            emitY();
            BAR();
            readY(0);
            if (__builtin_expect(i == 1, 0)) WAIT_V(9);
            WAIT_V(10);
            readA(0, 4, sn);
            writeP();
            { const int t = sa; sa = sn; sn = t; }
            po = (ps[0] + ps[1]) + (ps[2] + ps[3]);
            WAIT_L0();
            BAR();
        }
        readY(4); emitY();
    }
    if (remote && !late) STAMP_AT(3);
    if (!aux) call = __builtin_amdgcn_readfirstlane(__hip_atomic_load(cnt, RLX_AGT)) + 1u;

    WAIT_V(0);
    BAR();
    if (late) STAMP_AT(9); else if (remote) STAMP_AT(4); else STAMP_AT(10);
    if (wv != 0) return;
    if (lane == 0) {
        if (dst != rank) __hip_atomic_store((unsigned*)(P.ws[lab] + WS_FLAG) + rank * 32 + ti, call, RLX_SYS);
        __hip_atomic_store(cnt, call, RLX_AGT);
    }
    if (!SOLO && !remote && ti == 31) {
        GAS char* fl = myws + WS_FLAG;
        const bool skip = (lane >> 3) == rank || (lane & 7) >= 4;
        const unsigned long long t0 = __builtin_amdgcn_s_memrealtime();
        for (;;) {
            u32x4 f;
            asm volatile("s_nop 4\n\tglobal_load_dwordx4 %0, %1, %2 sc0 sc1\n\ts_waitcnt vmcnt(0)" : "=v"(f) : "v"(lane * 16), "s"(fl) : "memory");
            const bool ok = skip || (f.x >= call && f.y >= call && f.z >= call && f.w >= call);
            if (__all(ok)) break;
            if (__any(lane == rank * 8 && f.x == call) || __builtin_amdgcn_s_memrealtime() - t0 > DEADLINE_TICKS) {
                if (lane == 0) {
                    __hip_atomic_store(tmo, call, RLX_AGT);
                    __hip_atomic_store(P.myout + (size_t)rank * MLOC * NLOC, __builtin_nanf(""), RLX_SYS);
                }
                break;
            }
            __builtin_amdgcn_s_sleep(1);
        }
        STAMP_AT(5);
    }
    STAMP_OUT();
}

extern "C" void kernel_launch(int rank, int nranks, ncclComm_t comm,
                              hipStream_t stream,
                              void* const* d_in, const int* in_sizes, int n_in,
                              void* d_out, int out_size,
                              const int* in_dtypes, int out_dtype,
                              void* const* d_out_all,
                              void* const* d_ws_all, size_t ws_bytes) {
    if (nranks != NR || (unsigned)rank >= (unsigned)NR || n_in < 4 || ws_bytes < WS_NEED ||
        in_sizes[0] != MLOC * KDIM || in_sizes[1] != KDIM * NDIM || out_size != NR * MLOC * NLOC) {
        fprintf(stderr, "kernel_launch: unexpected problem shape\n");
        return;
    }
    static bool attr_set = false;
    if (!attr_set) {
        HIP_CHECK(hipFuncSetAttribute((const void*)gemm_a2a, hipFuncAttributeMaxDynamicSharedMemorySize, LDS_BYTES));
        attr_set = true;
    }
    Params P;
    P.x = (const uint8_t*)d_in[0];
    P.w = (const uint8_t*)d_in[1];
    P.sx = (const float*)d_in[2];
    P.sw = (const float*)d_in[3];
    for (int p = 0; p < NR; ++p) { P.out[p] = (float*)d_out_all[p]; P.ws[p] = (char*)d_ws_all[p]; }
    P.myws = P.ws[rank];
    P.myout = P.out[rank];
    P.rank = rank;
    {
        const int half[2][4] = {{0, 2, 3, 6}, {1, 4, 5, 7}};
        const int h = (rank == 1 || rank == 4 || rank == 5 || rank == 7);
        P.lmap = 0;
        for (int j = 0; j < 8; ++j) P.lmap |= (unsigned)(TOPO ? half[j < 4 ? h : 1 - h][j & 3] : j) << (4 * j);
    }
    hipLaunchKernelGGL(gemm_a2a, dim3(NWG), dim3(512), LDS_BYTES, stream, P);
    HIP_CHECK(hipGetLastError());
}
```

```cpp
#include <hip/hip_runtime.h>
#include <rccl/rccl.h>
#include <cstdio>
#include <cstdint>

#define HIP_CHECK(expr) do { hipError_t _e = (expr); if (_e != hipSuccess) { \
    fprintf(stderr, "HIP error %s at %s:%d\n", hipGetErrorString(_e), __FILE__, __LINE__); \
    return; } } while (0)

#define XFB 6
#define NL23 12
#define NEARLY 5
#define PFGAP 100
#define PFGATE 550
#define AMID 3
#define TOPO 1
#define SOLO 0

typedef int i32x8 __attribute__((ext_vector_type(8)));
typedef int i32x4 __attribute__((ext_vector_type(4)));
typedef int i32x2 __attribute__((ext_vector_type(2)));
typedef float f32x4 __attribute__((ext_vector_type(4)));
typedef unsigned u32x4 __attribute__((ext_vector_type(4)));
#define LAS __attribute__((address_space(3)))
#define GAS __attribute__((address_space(1)))

constexpr int NR = 8;
constexpr int KDIM = 4096, NDIM = 4096, MLOC = 512, NLOC = 512;
constexpr int NBLK = MLOC / 16;
constexpr int SBLK = NBLK - XFB;
constexpr int NPUSH = 4;
constexpr int PUSH_WIN = 4, PUSH_INFL = 3;
constexpr int WAVE_LDS = 20480;
constexpr int LDS_BYTES = 8 * WAVE_LDS;
constexpr int NWG = 256;
static_assert(XFB >= 2 && XFB <= 8 && NL23 >= 2 && NL23 % 2 == 0 && 2 * NEARLY <= SBLK, "block split");

constexpr size_t WS_FLAG = 0;
constexpr size_t WS_XFLAG = 2048;
constexpr size_t WS_CNT = 4096;
constexpr size_t WS_XBUF = 1u << 20;
constexpr size_t XSLOT = (size_t)XFB * 16 * KDIM;
constexpr size_t WS_NEED = WS_XBUF + 2 * NR * XSLOT;
constexpr unsigned long long DEADLINE_TICKS = 50000000ull;

#define STAMP_AT(k) do {} while (0)
#define STAMP_OUT() do {} while (0)

struct Params {
    const uint8_t* x;
    const uint8_t* w;
    char* myws;
    float* myout;
    const float* sx;
    const float* sw;
    int rank;
    unsigned lmap;
    float* out[NR];
    char* ws[NR];
};

#define STR_(x) #x
#define STR(x) STR_(x)
#define WAIT_V(n) asm volatile("s_waitcnt vmcnt(" STR(n) ")" ::: "memory")
#define WAIT_VN(n) asm volatile("s_waitcnt vmcnt(%0)" :: "n"(n) : "memory")
#define WAIT_L0() do { __builtin_amdgcn_s_waitcnt(0xC07F); asm volatile("" ::: "memory"); } while (0)
#define BAR() asm volatile("s_barrier" ::: "memory")
#define SCHED() __builtin_amdgcn_sched_barrier(0)
#define RLX_SYS __ATOMIC_RELAXED, __HIP_MEMORY_SCOPE_SYSTEM
#define RLX_AGT __ATOMIC_RELAXED, __HIP_MEMORY_SCOPE_AGENT

__device__ __forceinline__ void st16_sys(void* p, f32x4 v) {
    asm volatile("global_store_dwordx4 %0, %1, off sc0 sc1\n\ts_nop 1" :: "v"(p), "v"(v) : "memory");
}
__device__ __forceinline__ void glds16(const uint8_t* base, unsigned voff, unsigned lds) {
    asm volatile("s_mov_b32 m0, %2\n\ts_nop 0\n\tglobal_load_lds_dwordx4 %0, %1" :: "v"(voff), "s"(base), "s"(lds) : "memory");
}
__device__ __forceinline__ void glds16_sys(const uint8_t* base, unsigned voff, unsigned lds) {
    asm volatile("s_mov_b32 m0, %2\n\ts_nop 0\n\tglobal_load_lds_dwordx4 %0, %1 sc0 sc1" :: "v"(voff), "s"(base), "s"(lds) : "memory");
}

template <int I> struct IC { static constexpr int value = I; };
template <int N, int I = 0, class F> __device__ __forceinline__ void static_for(F&& f) {
    if constexpr (I < N) { f(IC<I>{}); static_for<N, I + 1>(f); }
}
constexpr int push_wait(int i, int n, int w, int f) {
    int c = 0;
    if (i < f) { c = w - 1; for (int k = 0; k < i; ++k) c += (w + k < n); return c; }
    for (int k = i - f; k < i; ++k) c += (w + k < n) + (k > i - f);
    return c;
}

__global__ void __launch_bounds__(512, 2) gemm_a2a(Params P) {
    extern __shared__ __attribute__((aligned(16))) unsigned char smem[];
    const int lane = threadIdx.x & 63;
    const int wv = __builtin_amdgcn_readfirstlane(threadIdx.x >> 6);
    const int b = blockIdx.x;
    const int ti = 31 - (b >> 3);
    int rank = P.rank;
    GAS char* myws = (GAS char*)P.myws;
    const uint8_t* gx = P.x;
    const uint8_t* gw = P.w;
    unsigned lmap = P.lmap;
    asm volatile("" : "+s"(rank), "+s"(myws), "+s"(gx), "+s"(gw), "+s"(lmap));
    const int lab = (lmap >> ((b & 7) * 4)) & 7;
    GAS unsigned* const cnt = (GAS unsigned*)(myws + WS_CNT) + b;
    GAS unsigned* const tmo = (GAS unsigned*)(myws + WS_FLAG) + rank * 32;
    STAMP_AT(0);

    const int slice = ti & 7, grp = ti >> 3;
    const bool remote = lab != rank;
    const bool aux = ti >= 16;
    const bool late = remote && aux;
    const unsigned lbase = (unsigned)(uintptr_t)(LAS unsigned char*)smem + wv * WAVE_LDS;
    unsigned call = 0;

    if (aux) {
        const int q = ti - 16;
        unsigned prev;
        if (q >= 16 - NPUSH) {
            const unsigned long long tstart = __builtin_amdgcn_s_memrealtime();
            asm volatile("s_nop 4\n\tglobal_load_dword %0, %1, %2 sc1" : "=v"(prev) : "v"(0u), "s"(cnt) : "memory");
            if (remote) {
                constexpr int NX = XFB * 8 / NPUSH;
                const unsigned xoff = ((q - (16 - NPUSH)) * 8 + wv) * (NX * 1024) + lane * 16;
                const uint8_t* src = gx + (size_t)SBLK * 16 * KDIM;
                f32x4 v[NX];
#pragma unroll
                for (int i = 0; i < PUSH_WIN; ++i)
                    asm volatile("global_load_dwordx4 %0, %1, %2" : "=v"(v[i]) : "v"(xoff + i * 1024), "s"(src) : "memory");
                char* const pws = P.ws[lab];
                WAIT_VN(PUSH_WIN);
                asm volatile("" : "+v"(prev));
                call = __builtin_amdgcn_readfirstlane(prev) + 1u;
                char* d = pws + WS_XBUF + ((call & 1u) * NR + rank) * XSLOT + xoff;
                static_for<NX>([&](auto ic) {
                    constexpr int i = decltype(ic)::value;
                    WAIT_VN(push_wait(i, NX, PUSH_WIN, PUSH_INFL));
                    asm volatile("" : "+v"(v[i]));
                    st16_sys(d + i * 1024, v[i]);
                    if constexpr (PUSH_WIN + i < NX)
                        asm volatile("global_load_dwordx4 %0, %1, %2" : "=v"(v[PUSH_WIN + i]) : "v"(xoff + (PUSH_WIN + i) * 1024), "s"(src) : "memory");
                    if (i == 0) STAMP_AT(6);
                });
                WAIT_V(0);
                BAR();
                STAMP_AT(7);
                if (threadIdx.x == 0)
                    __hip_atomic_store((unsigned*)(pws + WS_XFLAG) + rank * 32 + (q - (16 - NPUSH)), call, RLX_SYS);
            }
            while (__builtin_amdgcn_s_memrealtime() - tstart < PFGATE) __builtin_amdgcn_s_sleep(2);
        } else {
            constexpr int NPW = (16 - NPUSH) * 8;
            const unsigned vl = lane * 16;
            const unsigned vp = (lane >> 5) * NDIM + (lane & 31) * 16;
            const int u = q * 8 + wv;
            auto pfB = [&](const uint8_t* base, int m) {
                const int row = ((m >> 8) & 3) * 1024 + ((m >> 10) * 4 + ((m >> 5) & 3)) * 128 + ((m >> 7) & 1) * 64 + (m & 31) * 2;
                glds16(base + (size_t)row * NDIM, vp, lbase);
            };
#pragma unroll 1
            for (int m = q * 4 + wv; wv < 4 && m < 128; m += NPW / 2) glds16(gx + m * 1024, vl, lbase);
            const uint8_t* wl = gw + lab * NLOC;
#pragma unroll 1
            for (int m = q * 4 + (wv & 3); m < 1024; m += NPW / 2) { pfB(wl, m + (wv >> 2) * 1024); WAIT_V(12); }
#pragma unroll 1
            for (int m = 128 + q * 4 + wv; wv < 4 && m < AMID * 128; m += NPW / 2) { glds16(gx + m * 1024, vl, lbase); WAIT_V(12); }
            asm volatile("s_nop 4\n\tglobal_load_dword %0, %1, %2 sc1" : "=v"(prev) : "v"(0u), "s"(cnt) : "memory");
            BAR();
            STAMP_AT(12);
#pragma unroll 1
            for (int m = AMID * 128 + u; m < NEARLY * 128; m += NPW) glds16(gx + m * 1024, vl, lbase);
            WAIT_V(0);
            STAMP_AT(13);
            const unsigned long long t1 = __builtin_amdgcn_s_memrealtime();
            while (__builtin_amdgcn_s_memrealtime() - t1 < PFGAP) __builtin_amdgcn_s_sleep(2);
            const int mend = (remote ? SBLK : NBLK) * 64;
#pragma unroll 1
            for (int m = NEARLY * 128 + u; m < mend; m += NPW) { glds16(gx + m * 1024, vl, lbase); WAIT_V(12); }
            if (remote) {
                BAR();
                const uint8_t* wo = gw + rank * NLOC;
#pragma unroll 1
                for (int m = u; m < 2048; m += NPW) { pfB(wo, m); WAIT_V(12); }
            }
        }
        WAIT_V(0);
        asm volatile("" : "+v"(prev));
        call = __builtin_amdgcn_readfirstlane(prev) + 1u;
    }

    const int dst = late ? rank : lab;
    int nblk, blk0;
    {
        const int nmain = remote ? SBLK : NBLK - NL23;
        if (late) { nblk = (XFB + 1 - (grp & 1)) >> 1; blk0 = grp & 1; }
        else if (!aux) { nblk = (nmain + 1 - grp) >> 1; blk0 = grp; }
        else { nblk = NL23 / 2; blk0 = nmain + (grp & 1); }
    }
    const uint8_t* abase = gx + (size_t)blk0 * 16 * KDIM;
    const int orow0 = late ? lab * MLOC + (SBLK + blk0) * 16 : rank * MLOC + blk0 * 16;
    constexpr int astep = 2 * 16 * KDIM, ostep = 2 * 16 * NLOC * 4;
    bool bad = false;
    if (late) {
        if (wv == 0) {
            GAS unsigned* fl = (GAS unsigned*)(myws + WS_XFLAG) + lab * 32 + (lane & (NPUSH - 1));
            const unsigned long long t0 = __builtin_amdgcn_s_memrealtime();
            while (!SOLO && !__all(__hip_atomic_load(fl, RLX_SYS) >= call)) {
                if (__hip_atomic_load(tmo, RLX_AGT) == call || __builtin_amdgcn_s_memrealtime() - t0 > DEADLINE_TICKS) {
                    __hip_atomic_store(tmo, call, RLX_AGT);
                    break;
                }
                __builtin_amdgcn_s_sleep(8);
            }
            WAIT_V(0);
        }
        BAR();
        bad = __builtin_amdgcn_readfirstlane(__hip_atomic_load(tmo, RLX_AGT)) == call;
        if (!bad) abase = (const uint8_t*)(char*)myws + WS_XBUF + ((call & 1u) * NR + lab) * XSLOT + (size_t)blk0 * 16 * KDIM;
        STAMP_AT(8);
    }

    unsigned char* const lw = smem + wv * WAVE_LDS;
    const int r8 = lane >> 3, c8 = lane & 7;
    const unsigned va0 = r8 * KDIM + ((c8 ^ (r8 >> 1)) << 4);
    const unsigned va1 = (8 + r8) * KDIM + ((c8 ^ (4 + (r8 >> 1))) << 4);
    const uint8_t* aw = abase + wv * 128;
    auto loadA = [&](int blk, int so) {
        const uint8_t* p = aw + (blk < nblk ? blk : nblk - 1) * astep;
#pragma unroll
        for (int ks = 0; ks < 4; ++ks) {
            glds16_sys(p + ks * 1024, va0, lbase + so + ks * 2048);
            glds16_sys(p + ks * 1024, va1, lbase + so + ks * 2048 + 1024);
        }
    };
    const int r16 = lane >> 2, c4 = lane & 3;
    const unsigned vb0 = r16 * NDIM + ((c4 ^ ((r16 >> 2) & 1)) << 4);
    const unsigned vb1 = (16 + r16) * NDIM + ((c4 ^ (((r16 >> 2) & 1) | 2)) << 4);
    const uint8_t* bw = gw + (size_t)wv * 128 * NDIM + dst * NLOC + slice * 64;
    auto loadB = [&](int gI, int slot) {
        const uint8_t* p = bw + (size_t)((gI >> 1) * 1024 + (gI & 1) * 64) * NDIM;
        const unsigned l = lbase + slot * 4096;
        glds16(p, vb0, l); glds16(p, vb1, l + 1024);
        glds16(p + 32 * NDIM, vb0, l + 2048); glds16(p + 32 * NDIM, vb1, l + 3072);
    };
    loadA(0, 0);
    loadB(0, 2); loadB(1, 3); loadB(2, 4);

    float scx, scw, scale;
    asm volatile("s_load_dword %0, %2, 0x0\n\ts_load_dword %1, %3, 0x0" : "=&s"(scx), "=&s"(scw) : "s"(P.sx), "s"(P.sw) : "memory");
    const int i16 = lane & 15, g = lane >> 4;
    const int ra0 = i16 * 128 + ((g ^ (i16 >> 1)) << 4);
    const int ra1 = i16 * 128 + (((4 + g) ^ (i16 >> 1)) << 4);
    i32x8 xa[4];
    auto readA = [&](int k0, int k1, int so) {
#pragma unroll
        for (int ks = k0; ks < k1; ++ks) {
            const unsigned char* p = lw + so + ks * 2048;
            const i32x4 t0 = *(const i32x4*)(p + ra0), t1 = *(const i32x4*)(p + ra1);
            i32x8& d = xa[ks];
            d[0] = t0.x; d[1] = t0.y; d[2] = t0.z; d[3] = t0.w; d[4] = t1.x; d[5] = t1.y; d[6] = t1.z; d[7] = t1.w;
        }
    };
    const int q8 = i16 >> 1, pp = lane & 1;
    const int fB = ((q8 >> 2) & 1) | ((g & 1) << 1);
    int rb[4];
#pragma unroll
    for (int t = 0; t < 4; ++t) rb[t] = g * 1024 + q8 * 64 + ((t ^ fB) << 4) + pp * 8;
    i32x8 wb[4][4];
    auto readB = [&](int gI, int slot) {
        const int ks = gI >> 1, h = gI & 1;
#pragma unroll
        for (int t = 0; t < 4; ++t) {
#pragma unroll
            for (int r = 0; r < 2; ++r) {
                const i32x2 v = __builtin_amdgcn_ds_read_tr8_b64_v2i32((LAS i32x2*)(lw + slot * 4096 + rb[t] + r * 512));
                wb[ks][t][4 * h + 2 * r] = v.x; wb[ks][t][4 * h + 2 * r + 1] = v.y;
            }
        }
    };
    f32x4 acc[4];
    auto mm = [&](int ks, int t0, int t1) {
#pragma unroll
        for (int t = t0; t < t1; ++t) {
            const f32x4 c = ks == 0 ? f32x4{0.f, 0.f, 0.f, 0.f} : acc[t];
            acc[t] = __builtin_amdgcn_mfma_scale_f32_16x16x128_f8f6f4(wb[ks][t], xa[ks], c, 1, 0, 0, 0, 0, 0);
        }
    };
    const bool young = wv >= 4;
    unsigned char* const pw = lw + 4 * 4096 + i16 * 256;
    const int yrow = 4 * (wv & 3) + g;
    const unsigned char* const py = smem + 4 * 4096 + yrow * 256 + ((i16 ^ (yrow & 7)) << 4);
    char* optr = (char*)(late ? P.myout : P.out[lab]) + ((size_t)(orow0 + yrow) * NLOC + slice * 64 + i16 * 4) * 4;
    f32x4 ps[4], po;
    auto writeP = [&]() {
#pragma unroll
        for (int t = 0; t < 4; ++t) *(f32x4*)(pw + (((4 * t + g) ^ (i16 & 7)) << 4)) = acc[t];
    };
    auto readY = [&](int w0) {
#pragma unroll
        for (int j = 0; j < 4; ++j) ps[j] = *(const f32x4*)(py + (w0 + j) * WAVE_LDS);
    };
    auto emitY = [&]() {
        const f32x4 s = (po + ((ps[0] + ps[1]) + (ps[2] + ps[3]))) * scale;
        st16_sys(optr, s);
        optr += ostep;
    };

    WAIT_V(16); readA(0, 2, 0); WAIT_L0(); loadB(3, 0);
    WAIT_V(16); readA(2, 4, 0); WAIT_L0(); loadB(4, 1);
    asm volatile("" : "+s"(scx), "+s"(scw));
    scale = bad ? __builtin_nanf("") : scx * scw;
    WAIT_V(16); readB(0, 2); WAIT_L0(); loadB(5, 2);
    WAIT_V(16); readB(1, 3); WAIT_L0(); loadB(6, 3); SCHED(); mm(0, 0, 4); SCHED();
    WAIT_V(16); readB(2, 4); WAIT_L0(); loadB(7, 4);
    if (young) BAR();
    WAIT_V(16); readB(3, 0); SCHED(); mm(1, 0, 4); SCHED();
    WAIT_V(12); readB(4, 1);
    WAIT_V(8); readB(5, 2); WAIT_L0(); SCHED(); mm(2, 0, 4); SCHED();
    loadA(1, 0); WAIT_V(12);
    readB(6, 3);
    WAIT_V(8);
    readB(7, 4); WAIT_L0(); SCHED(); mm(3, 0, 4); SCHED();
    readY(0);
    writeP();
    loadA(2, 2 * 4096);
    WAIT_V(8);
    readA(0, 4, 0);
    po = (ps[0] + ps[1]) + (ps[2] + ps[3]);
    WAIT_L0();
    BAR();
    if (!aux && remote) STAMP_AT(11);

    int sa = 0, sn = 2 * 4096;
    auto phaseX = [&](int i, bool yng) {
        const uint8_t* p = aw + (i + 2 < nblk ? i + 2 : nblk - 1) * astep;
        const unsigned l = lbase + sa;
#pragma unroll
        for (int ks = 0; ks < 4; ++ks) {
            SCHED();
            mm(ks, 0, 2);
            SCHED();
            glds16_sys(p + ks * 1024, va0, l + ks * 2048);
            SCHED();
            mm(ks, 2, 4);
            SCHED();
            glds16_sys(p + ks * 1024, va1, l + ks * 2048 + 1024);
            if (yng && ks == 2) readY(4);
        }
        SCHED();
    };
    if (!young) {
#pragma unroll 1
        for (int i = 1; i < nblk; ++i) {
            phaseX(i, false);
            BAR();
            WAIT_V(8);
            readA(0, 4, sn);
            writeP();
            { const int t = sa; sa = sn; sn = t; }
            WAIT_L0();
            BAR();
        }
        BAR();
    } else {
#pragma unroll 1
        for (int i = 1; i < nblk; ++i) {
            phaseX(i, true);
            emitY();
            BAR();
            readY(0);
            if (__builtin_expect(i == 1, 0)) WAIT_V(9);
            WAIT_V(10);
            readA(0, 4, sn);
            writeP();
            { const int t = sa; sa = sn; sn = t; }
            po = (ps[0] + ps[1]) + (ps[2] + ps[3]);
            WAIT_L0();
            BAR();
        }
        readY(4); emitY();
    }
    if (remote && !late) STAMP_AT(3);
    if (!aux) call = __builtin_amdgcn_readfirstlane(__hip_atomic_load(cnt, RLX_AGT)) + 1u;

    WAIT_V(0);
    BAR();
    if (late) STAMP_AT(9); else if (remote) STAMP_AT(4); else STAMP_AT(10);
    if (wv != 0) return;
    if (lane == 0) {
        if (dst != rank) __hip_atomic_store((unsigned*)(P.ws[lab] + WS_FLAG) + rank * 32 + ti, call, RLX_SYS);
        __hip_atomic_store(cnt, call, RLX_AGT);
    }
    if (!SOLO && !remote && ti == 31) {
        GAS char* fl = myws + WS_FLAG;
        const bool skip = (lane >> 3) == rank || (lane & 7) >= 4;
        const unsigned long long t0 = __builtin_amdgcn_s_memrealtime();
        for (;;) {
            u32x4 f;
            asm volatile("s_nop 4\n\tglobal_load_dwordx4 %0, %1, %2 sc0 sc1\n\ts_waitcnt vmcnt(0)" : "=v"(f) : "v"(lane * 16), "s"(fl) : "memory");
            const bool ok = skip || (f.x >= call && f.y >= call && f.z >= call && f.w >= call);
            if (__all(ok)) break;
            if (__any(lane == rank * 8 && f.x == call) || __builtin_amdgcn_s_memrealtime() - t0 > DEADLINE_TICKS) {
                if (lane == 0) {
                    __hip_atomic_store(tmo, call, RLX_AGT);
                    __hip_atomic_store(P.myout + (size_t)rank * MLOC * NLOC, __builtin_nanf(""), RLX_SYS);
                }
                break;
            }
            __builtin_amdgcn_s_sleep(1);
        }
        STAMP_AT(5);
    }
    STAMP_OUT();
}

extern "C" void kernel_launch(int rank, int nranks, ncclComm_t comm,
                              hipStream_t stream,
                              void* const* d_in, const int* in_sizes, int n_in,
                              void* d_out, int out_size,
                              const int* in_dtypes, int out_dtype,
                              void* const* d_out_all,
                              void* const* d_ws_all, size_t ws_bytes) {
    if (nranks != NR || (unsigned)rank >= (unsigned)NR || n_in < 4 || ws_bytes < WS_NEED ||
        in_sizes[0] != MLOC * KDIM || in_sizes[1] != KDIM * NDIM || out_size != NR * MLOC * NLOC) {
        fprintf(stderr, "kernel_launch: unexpected problem shape\n");
        return;
    }
    static bool attr_set = false;
    if (!attr_set) {
        HIP_CHECK(hipFuncSetAttribute((const void*)gemm_a2a, hipFuncAttributeMaxDynamicSharedMemorySize, LDS_BYTES));
        attr_set = true;
    }
    Params P;
    P.x = (const uint8_t*)d_in[0];
    P.w = (const uint8_t*)d_in[1];
    P.sx = (const float*)d_in[2];
    P.sw = (const float*)d_in[3];
    for (int p = 0; p < NR; ++p) { P.out[p] = (float*)d_out_all[p]; P.ws[p] = (char*)d_ws_all[p]; }
    P.myws = P.ws[rank];
    P.myout = P.out[rank];
    P.rank = rank;
    {
        const int half[2][4] = {{0, 2, 3, 6}, {1, 4, 5, 7}};
        const int h = (rank == 1 || rank == 4 || rank == 5 || rank == 7);
        P.lmap = 0;
        for (int j = 0; j < 8; ++j) P.lmap |= (unsigned)(TOPO ? half[j < 4 ? h : 1 - h][j & 3] : j) << (4 * j);
    }
    hipLaunchKernelGGL(gemm_a2a, dim3(NWG), dim3(512), LDS_BYTES, stream, P);
    HIP_CHECK(hipGetLastError());
}
```

```cpp
#include <hip/hip_runtime.h>
#include <rccl/rccl.h>
#include <cstdio>
#include <cstdint>

#define HIP_CHECK(expr) do { hipError_t _e = (expr); if (_e != hipSuccess) { \
    fprintf(stderr, "HIP error %s at %s:%d\n", hipGetErrorString(_e), __FILE__, __LINE__); \
    return; } } while (0)

#define XFB 6
#define NL23 12
#define NEARLY 5
#define PFGAP 100
#define PFGATE 550
#define AMID 3
#define TOPO 1
#define SOLO 0

typedef int i32x8 __attribute__((ext_vector_type(8)));
typedef int i32x4 __attribute__((ext_vector_type(4)));
typedef int i32x2 __attribute__((ext_vector_type(2)));
typedef float f32x4 __attribute__((ext_vector_type(4)));
typedef unsigned u32x4 __attribute__((ext_vector_type(4)));
#define LAS __attribute__((address_space(3)))
#define GAS __attribute__((address_space(1)))

constexpr int NR = 8;
constexpr int KDIM = 4096, NDIM = 4096, MLOC = 512, NLOC = 512;
constexpr int NBLK = MLOC / 16;
constexpr int SBLK = NBLK - XFB;
constexpr int NPUSH = 4;
constexpr int PUSH_WIN = 4, PUSH_INFL = 3;
constexpr int WAVE_LDS = 20480;
constexpr int LDS_BYTES = 8 * WAVE_LDS;
constexpr int NWG = 256;
static_assert(XFB >= 2 && XFB <= 8 && NL23 >= 2 && NL23 % 2 == 0 && 2 * NEARLY <= SBLK, "block split");

constexpr size_t WS_FLAG = 0;
constexpr size_t WS_XFLAG = 2048;
constexpr size_t WS_CNT = 4096;
constexpr size_t WS_SLOT = 16384;
constexpr size_t WS_XBUF = 1u << 20;
constexpr size_t XSLOT = (size_t)XFB * 16 * KDIM;
constexpr size_t WS_NEED = WS_XBUF + 2 * NR * XSLOT;
constexpr unsigned long long DEADLINE_TICKS = 50000000ull;

#define STAMP_AT(k) do {} while (0)
#define STAMP_OUT() do {} while (0)

struct Params {
    const uint8_t* x;
    const uint8_t* w;
    char* myws;
    float* myout;
    const float* sx;
    const float* sw;
    int rank;
    unsigned lmap;
    float* out[NR];
    char* ws[NR];
};

#define STR_(x) #x
#define STR(x) STR_(x)
#define WAIT_V(n) asm volatile("s_waitcnt vmcnt(" STR(n) ")" ::: "memory")
#define WAIT_VN(n) asm volatile("s_waitcnt vmcnt(%0)" :: "n"(n) : "memory")
#define WAIT_L0() do { __builtin_amdgcn_s_waitcnt(0xC07F); asm volatile("" ::: "memory"); } while (0)
#define BAR() asm volatile("s_barrier" ::: "memory")
#define SCHED() __builtin_amdgcn_sched_barrier(0)
#define RLX_SYS __ATOMIC_RELAXED, __HIP_MEMORY_SCOPE_SYSTEM
#define RLX_AGT __ATOMIC_RELAXED, __HIP_MEMORY_SCOPE_AGENT

__device__ __forceinline__ void st16_sys(void* p, f32x4 v) {
    asm volatile("global_store_dwordx4 %0, %1, off sc0 sc1\n\ts_nop 1" :: "v"(p), "v"(v) : "memory");
}
__device__ __forceinline__ void glds16(const uint8_t* base, unsigned voff, unsigned lds) {
    asm volatile("s_mov_b32 m0, %2\n\ts_nop 0\n\tglobal_load_lds_dwordx4 %0, %1" :: "v"(voff), "s"(base), "s"(lds) : "memory");
}
__device__ __forceinline__ void glds16_sys(const uint8_t* base, unsigned voff, unsigned lds) {
    asm volatile("s_mov_b32 m0, %2\n\ts_nop 0\n\tglobal_load_lds_dwordx4 %0, %1 sc0 sc1" :: "v"(voff), "s"(base), "s"(lds) : "memory");
}

template <int I> struct IC { static constexpr int value = I; };
template <int N, int I = 0, class F> __device__ __forceinline__ void static_for(F&& f) {
    if constexpr (I < N) { f(IC<I>{}); static_for<N, I + 1>(f); }
}
constexpr int push_wait(int i, int n, int w, int f) {
    int c = 0;
    if (i < f) { c = w - 1; for (int k = 0; k < i; ++k) c += (w + k < n); return c; }
    for (int k = i - f; k < i; ++k) c += (w + k < n) + (k > i - f);
    return c;
}

__global__ void __launch_bounds__(512, 2) gemm_a2a(Params P) {
    extern __shared__ __attribute__((aligned(16))) unsigned char smem[];
    int label = __builtin_amdgcn_s_getreg((3 << 11) | 20) & 7;
    bool owned = false;
    STAMP_AT(0);
  for (;;) {
    int tid = threadIdx.x, b = blockIdx.x;
    int rank = P.rank;
    GAS char* myws = (GAS char*)P.myws;
    const uint8_t* gx = P.x;
    const uint8_t* gw = P.w;
    unsigned lmap = P.lmap;
    asm volatile("" : "+s"(rank), "+s"(myws), "+s"(gx), "+s"(gw), "+s"(lmap), "+s"(b), "+v"(tid));
    const int lane = tid & 63;
    const int wv = __builtin_amdgcn_readfirstlane(tid >> 6);
    const int ti = 31 - (b >> 3);
    GAS unsigned* const cnt = (GAS unsigned*)(myws + WS_CNT) + b;
    GAS unsigned* const tmo = (GAS unsigned*)(myws + WS_FLAG) + rank * 32;
    const int slice = ti & 7, grp = ti >> 3;
    const bool aux = ti >= 16;
    GAS unsigned* const slots = (GAS unsigned*)(myws + WS_SLOT) + ti * 8 * 32;
    auto reclaim = [&](unsigned call_) {
        int l = label;
#pragma unroll 1
        for (int k = 1; k < 8; ++k) {
            l = (label + k) & 7;
            unsigned e = call_ - 1u;
            if (lane == 0) __hip_atomic_compare_exchange_strong(slots + l * 32, &e, call_, __ATOMIC_RELAXED, RLX_AGT);
            if (__builtin_amdgcn_readfirstlane(e) == call_ - 1u) break;
        }
        return l;
    };
    const int lab = (lmap >> (label * 4)) & 7;
    const bool remote = lab != rank;
    const bool late = remote && aux;
    const unsigned lbase = (unsigned)(uintptr_t)(LAS unsigned char*)smem + wv * WAVE_LDS;
    unsigned call = 0, prevw = 0, seen = 0;

    if (aux) {
        const int q = ti - 16;
        unsigned prev;
        if (q >= 16 - NPUSH) {
            const unsigned long long tstart = __builtin_amdgcn_s_memrealtime();
            asm volatile("s_nop 4\n\tglobal_load_dword %0, %1, %2 sc1" : "=v"(prev) : "v"(0u), "s"(cnt) : "memory");
            if (remote) {
                constexpr int NX = XFB * 8 / NPUSH;
                const unsigned xoff = ((q - (16 - NPUSH)) * 8 + wv) * (NX * 1024) + lane * 16;
                const uint8_t* src = gx + (size_t)SBLK * 16 * KDIM;
                f32x4 v[NX];
#pragma unroll
                for (int i = 0; i < PUSH_WIN; ++i)
                    asm volatile("global_load_dwordx4 %0, %1, %2" : "=v"(v[i]) : "v"(xoff + i * 1024), "s"(src) : "memory");
                char* const pws = P.ws[lab];
                WAIT_VN(PUSH_WIN);
                asm volatile("" : "+v"(prev));
                call = __builtin_amdgcn_readfirstlane(prev) + 1u;
                char* d = pws + WS_XBUF + ((call & 1u) * NR + rank) * XSLOT + xoff;
                static_for<NX>([&](auto ic) {
                    constexpr int i = decltype(ic)::value;
                    WAIT_VN(push_wait(i, NX, PUSH_WIN, PUSH_INFL));
                    asm volatile("" : "+v"(v[i]));
                    st16_sys(d + i * 1024, v[i]);
                    if constexpr (PUSH_WIN + i < NX)
                        asm volatile("global_load_dwordx4 %0, %1, %2" : "=v"(v[PUSH_WIN + i]) : "v"(xoff + (PUSH_WIN + i) * 1024), "s"(src) : "memory");
                    if (i == 0) STAMP_AT(6);
                });
                WAIT_V(0);
                BAR();
                STAMP_AT(7);
                if (tid == 0)
                    __hip_atomic_store((unsigned*)(pws + WS_XFLAG) + rank * 32 + (q - (16 - NPUSH)), call, RLX_SYS);
            }
            while (__builtin_amdgcn_s_memrealtime() - tstart < PFGATE) __builtin_amdgcn_s_sleep(2);
        } else {
            constexpr int NPW = (16 - NPUSH) * 8;
            const unsigned vl = lane * 16;
            const unsigned vp = (lane >> 5) * NDIM + (lane & 31) * 16;
            const int u = q * 8 + wv;
            auto pfB = [&](const uint8_t* base, int m) {
                const int row = ((m >> 8) & 3) * 1024 + ((m >> 10) * 4 + ((m >> 5) & 3)) * 128 + ((m >> 7) & 1) * 64 + (m & 31) * 2;
                glds16(base + (size_t)row * NDIM, vp, lbase);
            };
#pragma unroll 1
            for (int m = q * 4 + wv; wv < 4 && m < 128; m += NPW / 2) glds16(gx + m * 1024, vl, lbase);
            const uint8_t* wl = gw + lab * NLOC;
#pragma unroll 1
            for (int m = q * 4 + (wv & 3); m < 1024; m += NPW / 2) { pfB(wl, m + (wv >> 2) * 1024); WAIT_V(12); }
#pragma unroll 1
            for (int m = 128 + q * 4 + wv; wv < 4 && m < AMID * 128; m += NPW / 2) { glds16(gx + m * 1024, vl, lbase); WAIT_V(12); }
            asm volatile("s_nop 4\n\tglobal_load_dword %0, %1, %2 sc1" : "=v"(prev) : "v"(0u), "s"(cnt) : "memory");
            BAR();
            STAMP_AT(12);
#pragma unroll 1
            for (int m = AMID * 128 + u; m < NEARLY * 128; m += NPW) glds16(gx + m * 1024, vl, lbase);
            WAIT_V(0);
            STAMP_AT(13);
            const unsigned long long t1 = __builtin_amdgcn_s_memrealtime();
            while (__builtin_amdgcn_s_memrealtime() - t1 < PFGAP) __builtin_amdgcn_s_sleep(2);
            const int mend = (remote ? SBLK : NBLK) * 64;
#pragma unroll 1
            for (int m = NEARLY * 128 + u; m < mend; m += NPW) { glds16(gx + m * 1024, vl, lbase); WAIT_V(12); }
            if (remote) {
                BAR();
                const uint8_t* wo = gw + rank * NLOC;
#pragma unroll 1
                for (int m = u; m < 2048; m += NPW) { pfB(wo, m); WAIT_V(12); }
            }
        }
        WAIT_V(0);
        asm volatile("" : "+v"(prev));
        call = __builtin_amdgcn_readfirstlane(prev) + 1u;
        if (!owned) {
            if (wv == 0) {
                unsigned e = call - 1u;
                if (lane == 0) __hip_atomic_compare_exchange_strong(slots + label * 32, &e, call, __ATOMIC_RELAXED, RLX_AGT);
                const int l = __builtin_amdgcn_readfirstlane(e) == call - 1u ? label : reclaim(call);
                *(volatile LAS int*)smem = l;
                WAIT_L0();
            }
            BAR();
            const int l = __builtin_amdgcn_readfirstlane(*(volatile LAS int*)smem);
            WAIT_L0();
            BAR();
            owned = true;
            if (l != label) { label = l; continue; }
        }
    } else {
        asm volatile("s_nop 4\n\tglobal_load_dword %0, %1, %2 sc1" : "=v"(prevw) : "v"(0u), "s"(cnt) : "memory");
    }

    const int dst = late ? rank : lab;
    int nblk, blk0;
    {
        const int nmain = remote ? SBLK : NBLK - NL23;
        if (late) { nblk = (XFB + 1 - (grp & 1)) >> 1; blk0 = grp & 1; }
        else if (!aux) { nblk = (nmain + 1 - grp) >> 1; blk0 = grp; }
        else { nblk = NL23 / 2; blk0 = nmain + (grp & 1); }
    }
    const uint8_t* abase = gx + (size_t)blk0 * 16 * KDIM;
    const int orow0 = late ? lab * MLOC + (SBLK + blk0) * 16 : rank * MLOC + blk0 * 16;
    constexpr int astep = 2 * 16 * KDIM, ostep = 2 * 16 * NLOC * 4;
    bool bad = false;
    if (late) {
        if (wv == 0) {
            GAS unsigned* fl = (GAS unsigned*)(myws + WS_XFLAG) + lab * 32 + (lane & (NPUSH - 1));
            const unsigned long long t0 = __builtin_amdgcn_s_memrealtime();
            while (!SOLO && !__all(__hip_atomic_load(fl, RLX_SYS) >= call)) {
                if (__hip_atomic_load(tmo, RLX_AGT) == call || __builtin_amdgcn_s_memrealtime() - t0 > DEADLINE_TICKS) {
                    __hip_atomic_store(tmo, call, RLX_AGT);
                    break;
                }
                __builtin_amdgcn_s_sleep(8);
            }
            WAIT_V(0);
        }
        BAR();
        bad = __builtin_amdgcn_readfirstlane(__hip_atomic_load(tmo, RLX_AGT)) == call;
        if (!bad) abase = (const uint8_t*)(char*)myws + WS_XBUF + ((call & 1u) * NR + lab) * XSLOT + (size_t)blk0 * 16 * KDIM;
        STAMP_AT(8);
    }

    unsigned char* const lw = smem + wv * WAVE_LDS;
    const int r8 = lane >> 3, c8 = lane & 7;
    const unsigned va0 = r8 * KDIM + ((c8 ^ (r8 >> 1)) << 4);
    const unsigned va1 = (8 + r8) * KDIM + ((c8 ^ (4 + (r8 >> 1))) << 4);
    const uint8_t* aw = abase + wv * 128;
    auto loadA = [&](int blk, int so) {
        const uint8_t* p = aw + (blk < nblk ? blk : nblk - 1) * astep;
#pragma unroll
        for (int ks = 0; ks < 4; ++ks) {
            glds16_sys(p + ks * 1024, va0, lbase + so + ks * 2048);
            glds16_sys(p + ks * 1024, va1, lbase + so + ks * 2048 + 1024);
        }
    };
    const int r16 = lane >> 2, c4 = lane & 3;
    const unsigned vb0 = r16 * NDIM + ((c4 ^ ((r16 >> 2) & 1)) << 4);
    const unsigned vb1 = (16 + r16) * NDIM + ((c4 ^ (((r16 >> 2) & 1) | 2)) << 4);
    const uint8_t* bw = gw + (size_t)wv * 128 * NDIM + dst * NLOC + slice * 64;
    auto loadB = [&](int gI, int slot) {
        const uint8_t* p = bw + (size_t)((gI >> 1) * 1024 + (gI & 1) * 64) * NDIM;
        const unsigned l = lbase + slot * 4096;
        glds16(p, vb0, l); glds16(p, vb1, l + 1024);
        glds16(p + 32 * NDIM, vb0, l + 2048); glds16(p + 32 * NDIM, vb1, l + 3072);
    };
    loadA(0, 0);
    loadB(0, 2); loadB(1, 3); loadB(2, 4);

    float scx, scw, scale;
    asm volatile("s_load_dword %0, %2, 0x0\n\ts_load_dword %1, %3, 0x0" : "=&s"(scx), "=&s"(scw) : "s"(P.sx), "s"(P.sw) : "memory");
    const int i16 = lane & 15, g = lane >> 4;
    const int ra0 = i16 * 128 + ((g ^ (i16 >> 1)) << 4);
    const int ra1 = i16 * 128 + (((4 + g) ^ (i16 >> 1)) << 4);
    i32x8 xa[4];
    auto readA = [&](int k0, int k1, int so) {
#pragma unroll
        for (int ks = k0; ks < k1; ++ks) {
            const unsigned char* p = lw + so + ks * 2048;
            const i32x4 t0 = *(const i32x4*)(p + ra0), t1 = *(const i32x4*)(p + ra1);
            i32x8& d = xa[ks];
            d[0] = t0.x; d[1] = t0.y; d[2] = t0.z; d[3] = t0.w; d[4] = t1.x; d[5] = t1.y; d[6] = t1.z; d[7] = t1.w;
        }
    };
    const int q8 = i16 >> 1, pp = lane & 1;
    const int fB = ((q8 >> 2) & 1) | ((g & 1) << 1);
    int rb[4];
#pragma unroll
    for (int t = 0; t < 4; ++t) rb[t] = g * 1024 + q8 * 64 + ((t ^ fB) << 4) + pp * 8;
    i32x8 wb[4][4];
    auto readB = [&](int gI, int slot) {
        const int ks = gI >> 1, h = gI & 1;
#pragma unroll
        for (int t = 0; t < 4; ++t) {
#pragma unroll
            for (int r = 0; r < 2; ++r) {
                const i32x2 v = __builtin_amdgcn_ds_read_tr8_b64_v2i32((LAS i32x2*)(lw + slot * 4096 + rb[t] + r * 512));
                wb[ks][t][4 * h + 2 * r] = v.x; wb[ks][t][4 * h + 2 * r + 1] = v.y;
            }
        }
    };
    f32x4 acc[4];
    auto mm = [&](int ks, int t0, int t1) {
#pragma unroll
        for (int t = t0; t < t1; ++t) {
            const f32x4 c = ks == 0 ? f32x4{0.f, 0.f, 0.f, 0.f} : acc[t];
            acc[t] = __builtin_amdgcn_mfma_scale_f32_16x16x128_f8f6f4(wb[ks][t], xa[ks], c, 1, 0, 0, 0, 0, 0);
        }
    };
    const bool young = wv >= 4;
    unsigned char* const pw = lw + 4 * 4096 + i16 * 256;
    const int yrow = 4 * (wv & 3) + g;
    const unsigned char* const py = smem + 4 * 4096 + yrow * 256 + ((i16 ^ (yrow & 7)) << 4);
    char* optr = (char*)(late ? P.myout : P.out[lab]) + ((size_t)(orow0 + yrow) * NLOC + slice * 64 + i16 * 4) * 4;
    f32x4 ps[4], po;
    auto writeP = [&]() {
#pragma unroll
        for (int t = 0; t < 4; ++t) *(f32x4*)(pw + (((4 * t + g) ^ (i16 & 7)) << 4)) = acc[t];
    };
    auto readY = [&](int w0) {
#pragma unroll
        for (int j = 0; j < 4; ++j) ps[j] = *(const f32x4*)(py + (w0 + j) * WAVE_LDS);
    };
    auto emitY = [&]() {
        const f32x4 s = (po + ((ps[0] + ps[1]) + (ps[2] + ps[3]))) * scale;
        st16_sys(optr, s);
        optr += ostep;
    };

    WAIT_V(16); readA(0, 2, 0); WAIT_L0(); loadB(3, 0);
    WAIT_V(16); readA(2, 4, 0); WAIT_L0(); loadB(4, 1);
    asm volatile("" : "+s"(scx), "+s"(scw), "+v"(prevw));
    if (!aux) call = __builtin_amdgcn_readfirstlane(prevw) + 1u;
    scale = bad ? __builtin_nanf("") : scx * scw;
    WAIT_V(16); readB(0, 2); WAIT_L0(); loadB(5, 2);
    WAIT_V(16); readB(1, 3); WAIT_L0(); loadB(6, 3); SCHED(); mm(0, 0, 4); SCHED();
    WAIT_V(16); readB(2, 4); WAIT_L0(); loadB(7, 4);
    if (young) BAR();
    WAIT_V(16); readB(3, 0); SCHED(); mm(1, 0, 4); SCHED();
    WAIT_V(12); readB(4, 1);
    WAIT_V(8); readB(5, 2); WAIT_L0(); SCHED(); mm(2, 0, 4); SCHED();
    loadA(1, 0); WAIT_V(12);
    readB(6, 3);
    WAIT_V(8);
    readB(7, 4); WAIT_L0(); SCHED(); mm(3, 0, 4); SCHED();
    readY(0);
    writeP();
    loadA(2, 2 * 4096);
    WAIT_V(8);
    readA(0, 4, 0);
    po = (ps[0] + ps[1]) + (ps[2] + ps[3]);
    WAIT_L0();
    BAR();
    if (!aux && remote) STAMP_AT(11);

    int sa = 0, sn = 2 * 4096;
    auto phaseX = [&](int i, bool yng) {
        const uint8_t* p = aw + (i + 2 < nblk ? i + 2 : nblk - 1) * astep;
        const unsigned l = lbase + sa;
#pragma unroll
        for (int ks = 0; ks < 4; ++ks) {
            SCHED();
            mm(ks, 0, 2);
            SCHED();
            glds16_sys(p + ks * 1024, va0, l + ks * 2048);
            SCHED();
            mm(ks, 2, 4);
            SCHED();
            glds16_sys(p + ks * 1024, va1, l + ks * 2048 + 1024);
            if (yng && ks == 2) readY(4);
        }
        SCHED();
    };
    if (!young) {
#pragma unroll 1
        for (int i = 1; i < nblk; ++i) {
            phaseX(i, false);
            BAR();
            WAIT_V(8);
            if (__builtin_expect(i == 2 && wv == 0 && !owned, 0)) {
                const i32x2 cv = {(int)call, (int)(call - 1u)};
                if (lane == 0)
                    asm volatile("s_nop 4\n\tglobal_atomic_cmpswap %0, %1, %2, %3 sc0" : "+v"(seen) : "v"(0u), "v"(cv), "s"(slots + label * 32) : "memory");
            }
            readA(0, 4, sn);
            writeP();
            { const int t = sa; sa = sn; sn = t; }
            WAIT_L0();
            BAR();
        }
        BAR();
    } else {
#pragma unroll 1
        for (int i = 1; i < nblk; ++i) {
            phaseX(i, true);
            emitY();
            BAR();
            readY(0);
            if (__builtin_expect(i == 1, 0)) WAIT_V(9);
            WAIT_V(10);
            readA(0, 4, sn);
            writeP();
            { const int t = sa; sa = sn; sn = t; }
            po = (ps[0] + ps[1]) + (ps[2] + ps[3]);
            WAIT_L0();
            BAR();
        }
        readY(4); emitY();
    }
    if (remote && !late) STAMP_AT(3);

    WAIT_V(0);
    if (!owned && wv == 0) {
        asm volatile("" : "+v"(seen));
        *(volatile LAS int*)smem = __builtin_amdgcn_readfirstlane(seen) == call - 1u ? label : reclaim(call);
        WAIT_L0();
    }
    BAR();
    if (tid == 0 && dst != rank) __hip_atomic_store((unsigned*)(P.ws[lab] + WS_FLAG) + rank * 32 + ti, call, RLX_SYS);
    if (!owned) {
        const int l = __builtin_amdgcn_readfirstlane(*(volatile LAS int*)smem);
        owned = true;
        if (l != label) { WAIT_L0(); BAR(); label = l; continue; }
    }
    if (late) STAMP_AT(9); else if (remote) STAMP_AT(4); else STAMP_AT(10);
    if (wv != 0) return;
    if (lane == 0) __hip_atomic_store(cnt, call, RLX_AGT);
    if (!SOLO && !remote && ti == 31) {
        GAS char* fl = myws + WS_FLAG;
        const bool skip = (lane >> 3) == rank || (lane & 7) >= 4;
        const unsigned long long t0 = __builtin_amdgcn_s_memrealtime();
        for (;;) {
            u32x4 f;
            asm volatile("s_nop 4\n\tglobal_load_dwordx4 %0, %1, %2 sc0 sc1\n\ts_waitcnt vmcnt(0)" : "=v"(f) : "v"(lane * 16), "s"(fl) : "memory");
            const bool ok = skip || (f.x >= call && f.y >= call && f.z >= call && f.w >= call);
            if (__all(ok)) break;
            if (__any(lane == rank * 8 && f.x == call) || __builtin_amdgcn_s_memrealtime() - t0 > DEADLINE_TICKS) {
                if (lane == 0) {
                    __hip_atomic_store(tmo, call, RLX_AGT);
                    __hip_atomic_store(P.myout + (size_t)rank * MLOC * NLOC, __builtin_nanf(""), RLX_SYS);
                }
                break;
            }
            __builtin_amdgcn_s_sleep(1);
        }
        STAMP_AT(5);
    }
    STAMP_OUT();
    return;
  }
}

extern "C" void kernel_launch(int rank, int nranks, ncclComm_t comm,
                              hipStream_t stream,
                              void* const* d_in, const int* in_sizes, int n_in,
                              void* d_out, int out_size,
                              const int* in_dtypes, int out_dtype,
                              void* const* d_out_all,
                              void* const* d_ws_all, size_t ws_bytes) {
    if (nranks != NR || (unsigned)rank >= (unsigned)NR || n_in < 4 || ws_bytes < WS_NEED ||
        in_sizes[0] != MLOC * KDIM || in_sizes[1] != KDIM * NDIM || out_size != NR * MLOC * NLOC) {
        fprintf(stderr, "kernel_launch: unexpected problem shape\n");
        return;
    }
    static bool attr_set = false;
    if (!attr_set) {
        HIP_CHECK(hipFuncSetAttribute((const void*)gemm_a2a, hipFuncAttributeMaxDynamicSharedMemorySize, LDS_BYTES));
        attr_set = true;
    }
    Params P;
    P.x = (const uint8_t*)d_in[0];
    P.w = (const uint8_t*)d_in[1];
    P.sx = (const float*)d_in[2];
    P.sw = (const float*)d_in[3];
    for (int p = 0; p < NR; ++p) { P.out[p] = (float*)d_out_all[p]; P.ws[p] = (char*)d_ws_all[p]; }
    P.myws = P.ws[rank];
    P.myout = P.out[rank];
    P.rank = rank;
    {
        const int half[2][4] = {{0, 2, 3, 6}, {1, 4, 5, 7}};
        const int h = (rank == 1 || rank == 4 || rank == 5 || rank == 7);
        P.lmap = 0;
        for (int j = 0; j < 8; ++j) P.lmap |= (unsigned)(TOPO ? half[j < 4 ? h : 1 - h][j & 3] : j) << (4 * j);
    }
    hipLaunchKernelGGL(gemm_a2a, dim3(NWG), dim3(512), LDS_BYTES, stream, P);
    HIP_CHECK(hipGetLastError());
}
```

```cpp
#include <hip/hip_runtime.h>
#include <rccl/rccl.h>
#include <cstdio>
#include <cstdint>

#define HIP_CHECK(expr) do { hipError_t _e = (expr); if (_e != hipSuccess) { \
    fprintf(stderr, "HIP error %s at %s:%d\n", hipGetErrorString(_e), __FILE__, __LINE__); \
    return; } } while (0)

constexpr int XFB = 6;
constexpr int NL23 = 12;
constexpr int PFGAP = 100;
constexpr int PFGATE = 550;
constexpr int AMID = 3;
constexpr int NEARLY = 5;

typedef int i32x8 __attribute__((ext_vector_type(8)));
typedef int i32x4 __attribute__((ext_vector_type(4)));
typedef int i32x2 __attribute__((ext_vector_type(2)));
typedef float f32x4 __attribute__((ext_vector_type(4)));
typedef unsigned u32x4 __attribute__((ext_vector_type(4)));
#define LAS __attribute__((address_space(3)))
#define GAS __attribute__((address_space(1)))

constexpr int NR = 8;
constexpr int KDIM = 4096, NDIM = 4096, MLOC = 512, NLOC = 512;
constexpr int NBLK = MLOC / 16;
constexpr int SBLK = NBLK - XFB;
constexpr int NPUSH = 4;
constexpr int PUSH_WIN = 3, PUSH_INFL = 2;
static_assert(XFB * 8 % NPUSH == 0 && PUSH_WIN > PUSH_INFL && PUSH_WIN <= XFB * 8 / NPUSH, "forwarding split");
constexpr int WAVE_LDS = 20480;
constexpr int LDS_BYTES = 8 * WAVE_LDS;
constexpr int NWG = 256;
static_assert(XFB >= 2 && XFB <= 8 && NL23 >= 2 && NL23 % 2 == 0 && 2 * NEARLY <= SBLK && NBLK - NL23 >= 6, "block split");

constexpr size_t WS_FLAG = 0;
constexpr size_t WS_XFLAG = 2048;
constexpr size_t WS_CNT = 4096;
constexpr size_t WS_SLOT = 16384;
constexpr size_t WS_XBUF = 1u << 20;
constexpr size_t XSLOT = (size_t)XFB * 16 * KDIM;
constexpr size_t WS_NEED = WS_XBUF + 2 * NR * XSLOT;
constexpr unsigned long long DEADLINE_TICKS = 50000000ull;

struct Params {
    const uint8_t* x;
    const uint8_t* w;
    char* myws;
    float* myout;
    const float* sx;
    const float* sw;
    int rank;
    unsigned lmap;
    float* out[NR];
    char* ws[NR];
};

#define STR_(x) #x
#define STR(x) STR_(x)
#define WAIT_V(n) asm volatile("s_waitcnt vmcnt(" STR(n) ")" ::: "memory")
#define WAIT_VN(n) asm volatile("s_waitcnt vmcnt(%0)" :: "n"(n) : "memory")
#define WAIT_L0() do { __builtin_amdgcn_s_waitcnt(0xC07F); asm volatile("" ::: "memory"); } while (0)
#define BAR() asm volatile("s_barrier" ::: "memory")
#define SCHED() __builtin_amdgcn_sched_barrier(0)
#define RLX_SYS __ATOMIC_RELAXED, __HIP_MEMORY_SCOPE_SYSTEM
#define RLX_AGT __ATOMIC_RELAXED, __HIP_MEMORY_SCOPE_AGENT

__device__ __forceinline__ void st16_sys(void* p, f32x4 v) {
    asm volatile("global_store_dwordx4 %0, %1, off sc0 sc1\n\ts_nop 1" :: "v"(p), "v"(v) : "memory");
}
__device__ __forceinline__ void ld16(f32x4& v, const uint8_t* base, unsigned voff) {
    asm volatile("global_load_dwordx4 %0, %1, %2" : "=v"(v) : "v"(voff), "s"(base) : "memory");
}
__device__ __forceinline__ void glds16(const uint8_t* base, unsigned voff, unsigned lds) {
    asm volatile("s_mov_b32 m0, %2\n\ts_nop 0\n\tglobal_load_lds_dwordx4 %0, %1" :: "v"(voff), "s"(base), "s"(lds) : "memory");
}
__device__ __forceinline__ void glds16_sys(const uint8_t* base, unsigned voff, unsigned lds) {
    asm volatile("s_mov_b32 m0, %2\n\ts_nop 0\n\tglobal_load_lds_dwordx4 %0, %1 sc0 sc1" :: "v"(voff), "s"(base), "s"(lds) : "memory");
}

template <int I> struct IC { static constexpr int value = I; };
template <int N, int I = 0, class F> __device__ __forceinline__ void static_for(F&& f) {
    if constexpr (I < N) { f(IC<I>{}); static_for<N, I + 1>(f); }
}
constexpr int push_wait(int i, int n, int w, int f) {
    int c = 0;
    if (i < f) { c = w - 1; for (int k = 0; k < i; ++k) c += (w + k < n); return c; }
    for (int k = i - f; k < i; ++k) c += (w + k < n) + (k > i - f);
    return c;
}

__global__ void __launch_bounds__(512, 2) gemm_a2a(Params P) {
    extern __shared__ __attribute__((aligned(16))) unsigned char smem[];
    int label = __builtin_amdgcn_s_getreg((3 << 11) | 20) & 7;
    bool owned = false;
  for (;;) {
    int tid = threadIdx.x, b = blockIdx.x;
    int rank = P.rank;
    GAS char* myws = (GAS char*)P.myws;
    const uint8_t* gx = P.x;
    const uint8_t* gw = P.w;
    unsigned lmap = P.lmap;
    asm volatile("" : "+s"(rank), "+s"(myws), "+s"(gx), "+s"(gw), "+s"(lmap), "+s"(b), "+v"(tid));
    const int lane = tid & 63;
    const int wv = __builtin_amdgcn_readfirstlane(tid >> 6);
    const int ti = 31 - (b >> 3);
    GAS unsigned* const cnt = (GAS unsigned*)(myws + WS_CNT) + b;
    GAS unsigned* const tmo = (GAS unsigned*)(myws + WS_FLAG) + rank * 32;
    const int slice = ti & 7, grp = ti >> 3;
    const bool aux = ti >= 16;
    GAS unsigned* const slots = (GAS unsigned*)(myws + WS_SLOT) + ti * 8 * 32;
    auto reclaim = [&](unsigned call_) {
        int l = label;
#pragma unroll 1
        for (int k = 1; k < 8; ++k) {
            l = (label + k) & 7;
            unsigned e = call_ - 1u;
            if (lane == 0) __hip_atomic_compare_exchange_strong(slots + l * 32, &e, call_, __ATOMIC_RELAXED, RLX_AGT);
            if (__builtin_amdgcn_readfirstlane(e) == call_ - 1u) break;
        }
        return l;
    };
    const int lab = (lmap >> (label * 4)) & 7;
    const bool remote = lab != rank;
    const bool late = remote && aux;
    const unsigned lbase = (unsigned)(uintptr_t)(LAS unsigned char*)smem + wv * WAVE_LDS;
    unsigned call = 0, prevw = 0, seen = 0;

    if (aux) {
        const int q = ti - 16;
        unsigned prev;
        if (q >= 16 - NPUSH) {
            const unsigned long long tstart = __builtin_amdgcn_s_memrealtime();
            asm volatile("s_nop 4\n\tglobal_load_dword %0, %1, %2 sc1" : "=v"(prev) : "v"(0u), "s"(cnt) : "memory");
            if (remote) {
                constexpr int NX = XFB * 8 / NPUSH;
                const unsigned xoff = ((q - (16 - NPUSH)) * 8 + wv) * (NX * 1024) + lane * 16;
                const uint8_t* src = gx + (size_t)SBLK * 16 * KDIM;
                f32x4 v[NX];
#pragma unroll
                for (int i = 0; i < PUSH_WIN; ++i) ld16(v[i], src, xoff + i * 1024);
                char* const pws = P.ws[lab];
                WAIT_VN(PUSH_WIN);
                asm volatile("" : "+v"(prev));
                call = __builtin_amdgcn_readfirstlane(prev) + 1u;
                char* d = pws + WS_XBUF + ((call & 1u) * NR + rank) * XSLOT + xoff;
                static_for<NX>([&](auto ic) {
                    constexpr int i = decltype(ic)::value;
                    WAIT_VN(push_wait(i, NX, PUSH_WIN, PUSH_INFL));
                    asm volatile("" : "+v"(v[i]));
                    st16_sys(d + i * 1024, v[i]);
                    if constexpr (PUSH_WIN + i < NX) ld16(v[PUSH_WIN + i], src, xoff + (PUSH_WIN + i) * 1024);
                });
                WAIT_V(0);
                BAR();
                if (tid == 0)
                    __hip_atomic_store((unsigned*)(pws + WS_XFLAG) + rank * 32 + (q - (16 - NPUSH)), call, RLX_SYS);
            }
            while (__builtin_amdgcn_s_memrealtime() - tstart < PFGATE) __builtin_amdgcn_s_sleep(2);
        } else {
            constexpr int NPW = (16 - NPUSH) * 8;
            const unsigned vl = lane * 16;
            const unsigned vp = (lane >> 5) * NDIM + (lane & 31) * 16;
            const int u = q * 8 + wv;
            auto pfB = [&](const uint8_t* base, int m) {
                const int row = ((m >> 8) & 3) * 1024 + ((m >> 10) * 4 + ((m >> 5) & 3)) * 128 + ((m >> 7) & 1) * 64 + (m & 31) * 2;
                glds16(base + (size_t)row * NDIM, vp, lbase);
            };
#pragma unroll 1
            for (int m = q * 4 + wv; wv < 4 && m < 128; m += NPW / 2) glds16(gx + m * 1024, vl, lbase);
            const uint8_t* wl = gw + lab * NLOC;
#pragma unroll 1
            for (int m = q * 4 + (wv & 3); m < 1024; m += NPW / 2) { pfB(wl, m + (wv >> 2) * 1024); WAIT_V(12); }
#pragma unroll 1
            for (int m = 128 + q * 4 + wv; wv < 4 && m < AMID * 128; m += NPW / 2) { glds16(gx + m * 1024, vl, lbase); WAIT_V(12); }
            asm volatile("s_nop 4\n\tglobal_load_dword %0, %1, %2 sc1" : "=v"(prev) : "v"(0u), "s"(cnt) : "memory");
            BAR();
#pragma unroll 1
            for (int m = AMID * 128 + u; m < NEARLY * 128; m += NPW) glds16(gx + m * 1024, vl, lbase);
            WAIT_V(0);
            const unsigned long long t1 = __builtin_amdgcn_s_memrealtime();
            while (__builtin_amdgcn_s_memrealtime() - t1 < PFGAP) __builtin_amdgcn_s_sleep(2);
            const int mend = (remote ? SBLK : NBLK) * 64;
#pragma unroll 1
            for (int m = NEARLY * 128 + u; m < mend; m += NPW) { glds16(gx + m * 1024, vl, lbase); WAIT_V(12); }
            if (remote) {
                BAR();
                const uint8_t* wo = gw + rank * NLOC;
#pragma unroll 1
                for (int m = u; m < 2048; m += NPW) { pfB(wo, m); WAIT_V(12); }
            }
        }
        WAIT_V(0);
        asm volatile("" : "+v"(prev));
        call = __builtin_amdgcn_readfirstlane(prev) + 1u;
        if (!owned) {
            if (wv == 0) {
                unsigned e = call - 1u;
                if (lane == 0) __hip_atomic_compare_exchange_strong(slots + label * 32, &e, call, __ATOMIC_RELAXED, RLX_AGT);
                const int l = __builtin_amdgcn_readfirstlane(e) == call - 1u ? label : reclaim(call);
                *(volatile LAS int*)smem = l;
                WAIT_L0();
            }
            BAR();
            const int l = __builtin_amdgcn_readfirstlane(*(volatile LAS int*)smem);
            WAIT_L0();
            BAR();
            owned = true;
            if (l != label) { label = l; continue; }
        }
    } else {
        asm volatile("s_nop 4\n\tglobal_load_dword %0, %1, %2 sc1" : "+v"(prevw) : "v"(0u), "s"(cnt) : "memory");
    }

    const int dst = late ? rank : lab;
    int nblk, blk0;
    {
        const int nmain = remote ? SBLK : NBLK - NL23;
        if (late) { nblk = (XFB + 1 - (grp & 1)) >> 1; blk0 = grp & 1; }
        else if (!aux) { nblk = (nmain + 1 - grp) >> 1; blk0 = grp; }
        else { nblk = NL23 / 2; blk0 = nmain + (grp & 1); }
    }
    const uint8_t* abase = gx + (size_t)blk0 * 16 * KDIM;
    const int orow0 = late ? lab * MLOC + (SBLK + blk0) * 16 : rank * MLOC + blk0 * 16;
    constexpr int astep = 2 * 16 * KDIM, ostep = 2 * 16 * NLOC * 4;
    bool bad = false;
    if (late) {
        if (wv == 0) {
            GAS unsigned* fl = (GAS unsigned*)(myws + WS_XFLAG) + lab * 32 + (lane < NPUSH ? lane : 0);
            const unsigned long long t0 = __builtin_amdgcn_s_memrealtime();
            while (!__all(__hip_atomic_load(fl, RLX_SYS) >= call)) {
                if (__hip_atomic_load(tmo, RLX_AGT) == call || __builtin_amdgcn_s_memrealtime() - t0 > DEADLINE_TICKS) {
                    __hip_atomic_store(tmo, call, RLX_AGT);
                    break;
                }
                __builtin_amdgcn_s_sleep(8);
            }
            WAIT_V(0);
        }
        BAR();
        bad = __builtin_amdgcn_readfirstlane(__hip_atomic_load(tmo, RLX_AGT)) == call;
        if (!bad) abase = (const uint8_t*)(char*)myws + WS_XBUF + ((call & 1u) * NR + lab) * XSLOT + (size_t)blk0 * 16 * KDIM;
    }

    unsigned char* const lw = smem + wv * WAVE_LDS;
    const int r8 = lane >> 3, c8 = lane & 7;
    const unsigned va0 = r8 * KDIM + ((c8 ^ (r8 >> 1)) << 4);
    const unsigned va1 = (8 + r8) * KDIM + ((c8 ^ (4 + (r8 >> 1))) << 4);
    const uint8_t* aw = abase + wv * 128;
    auto loadA = [&](int blk, int so) {
        const uint8_t* p = aw + (blk < nblk ? blk : nblk - 1) * astep;
#pragma unroll
        for (int ks = 0; ks < 4; ++ks) {
            glds16_sys(p + ks * 1024, va0, lbase + so + ks * 2048);
            glds16_sys(p + ks * 1024, va1, lbase + so + ks * 2048 + 1024);
        }
    };
    const int r16 = lane >> 2, c4 = lane & 3;
    const unsigned vb0 = r16 * NDIM + ((c4 ^ ((r16 >> 2) & 1)) << 4);
    const unsigned vb1 = (16 + r16) * NDIM + ((c4 ^ (((r16 >> 2) & 1) | 2)) << 4);
    const uint8_t* bw = gw + (size_t)wv * 128 * NDIM + dst * NLOC + slice * 64;
    auto loadB = [&](int gI, int slot) {
        const uint8_t* p = bw + (size_t)((gI >> 1) * 1024 + (gI & 1) * 64) * NDIM;
        const unsigned l = lbase + slot * 4096;
        glds16(p, vb0, l); glds16(p, vb1, l + 1024);
        glds16(p + 32 * NDIM, vb0, l + 2048); glds16(p + 32 * NDIM, vb1, l + 3072);
    };
    loadA(0, 0);
    loadB(0, 2); loadB(1, 3); loadB(2, 4);

    float scx, scw, scale;
    asm volatile("s_load_dword %0, %2, 0x0\n\ts_load_dword %1, %3, 0x0" : "=&s"(scx), "=&s"(scw) : "s"(P.sx), "s"(P.sw) : "memory");
    const int i16 = lane & 15, g = lane >> 4;
    const int ra0 = i16 * 128 + ((g ^ (i16 >> 1)) << 4);
    const int ra1 = i16 * 128 + (((4 + g) ^ (i16 >> 1)) << 4);
    i32x8 xa[4];
    auto readA = [&](int k0, int k1, int so) {
#pragma unroll
        for (int ks = k0; ks < k1; ++ks) {
            const unsigned char* p = lw + so + ks * 2048;
            const i32x4 t0 = *(const i32x4*)(p + ra0), t1 = *(const i32x4*)(p + ra1);
            i32x8& d = xa[ks];
            d[0] = t0.x; d[1] = t0.y; d[2] = t0.z; d[3] = t0.w; d[4] = t1.x; d[5] = t1.y; d[6] = t1.z; d[7] = t1.w;
        }
    };
    const int q8 = i16 >> 1, pp = lane & 1;
    const int fB = ((q8 >> 2) & 1) | ((g & 1) << 1);
    int rb[4];
#pragma unroll
    for (int t = 0; t < 4; ++t) rb[t] = g * 1024 + q8 * 64 + ((t ^ fB) << 4) + pp * 8;
    i32x8 wb[4][4];
    auto readB = [&](int gI, int slot) {
        const int ks = gI >> 1, h = gI & 1;
#pragma unroll
        for (int t = 0; t < 4; ++t) {
#pragma unroll
            for (int r = 0; r < 2; ++r) {
                const i32x2 v = __builtin_amdgcn_ds_read_tr8_b64_v2i32((LAS i32x2*)(lw + slot * 4096 + rb[t] + r * 512));
                wb[ks][t][4 * h + 2 * r] = v.x; wb[ks][t][4 * h + 2 * r + 1] = v.y;
            }
        }
    };
    f32x4 acc[4];
    auto mm = [&](int ks, int t0, int t1) {
#pragma unroll
        for (int t = t0; t < t1; ++t) {
            const f32x4 c = ks == 0 ? f32x4{0.f, 0.f, 0.f, 0.f} : acc[t];
            acc[t] = __builtin_amdgcn_mfma_scale_f32_16x16x128_f8f6f4(wb[ks][t], xa[ks], c, 1, 0, 0, 0, 0, 0);
        }
    };
    const bool young = wv >= 4;
    unsigned char* const pw = lw + 4 * 4096 + i16 * 256;
    const int yrow = 4 * (wv & 3) + g;
    const unsigned char* const py = smem + 4 * 4096 + yrow * 256 + ((i16 ^ (yrow & 7)) << 4);
    char* optr = (char*)(late ? P.myout : P.out[lab]) + ((size_t)(orow0 + yrow) * NLOC + slice * 64 + i16 * 4) * 4;
    f32x4 ps[4], po;
    auto writeP = [&]() {
#pragma unroll
        for (int t = 0; t < 4; ++t) *(f32x4*)(pw + (((4 * t + g) ^ (i16 & 7)) << 4)) = acc[t];
    };
    auto readY = [&](int w0) {
#pragma unroll
        for (int j = 0; j < 4; ++j) ps[j] = *(const f32x4*)(py + (w0 + j) * WAVE_LDS);
    };
    auto emitY = [&]() {
        const f32x4 s = (po + ((ps[0] + ps[1]) + (ps[2] + ps[3]))) * scale;
        st16_sys(optr, s);
        optr += ostep;
    };

    WAIT_V(16); readA(0, 2, 0); WAIT_L0(); loadB(3, 0);
    WAIT_V(16); readA(2, 4, 0); WAIT_L0(); loadB(4, 1);
    asm volatile("" : "+s"(scx), "+s"(scw), "+v"(prevw));
    if (!aux) call = __builtin_amdgcn_readfirstlane(prevw) + 1u;
    scale = bad ? __builtin_nanf("") : scx * scw;
    WAIT_V(16); readB(0, 2); WAIT_L0(); loadB(5, 2);
    WAIT_V(16); readB(1, 3); WAIT_L0(); loadB(6, 3); SCHED(); mm(0, 0, 4); SCHED();
    WAIT_V(16); readB(2, 4); WAIT_L0(); loadB(7, 4);
    if (young) BAR();
    WAIT_V(16); readB(3, 0); SCHED(); mm(1, 0, 4); SCHED();
    WAIT_V(12); readB(4, 1);
    WAIT_V(8); readB(5, 2); WAIT_L0(); SCHED(); mm(2, 0, 4); SCHED();
    loadA(1, 0); WAIT_V(12);
    readB(6, 3);
    WAIT_V(8);
    readB(7, 4); WAIT_L0(); SCHED(); mm(3, 0, 4); SCHED();
    readY(0);
    writeP();
    loadA(2, 2 * 4096);
    WAIT_V(8);
    readA(0, 4, 0);
    po = (ps[0] + ps[1]) + (ps[2] + ps[3]);
    WAIT_L0();
    BAR();

    int sa = 0, sn = 2 * 4096;
    auto phaseX = [&](int i, bool yng) {
        const uint8_t* p = aw + (i + 2 < nblk ? i + 2 : nblk - 1) * astep;
        const unsigned l = lbase + sa;
#pragma unroll
        for (int ks = 0; ks < 4; ++ks) {
            SCHED();
            mm(ks, 0, 2);
            SCHED();
            glds16_sys(p + ks * 1024, va0, l + ks * 2048);
            SCHED();
            mm(ks, 2, 4);
            SCHED();
            glds16_sys(p + ks * 1024, va1, l + ks * 2048 + 1024);
            if (yng && ks == 2) readY(4);
        }
        SCHED();
    };
    if (!young) {
#pragma unroll 1
        for (int i = 1; i < nblk; ++i) {
            phaseX(i, false);
            BAR();
            WAIT_V(8);
            if (__builtin_expect(i == 2 && wv == 0 && !owned, 0)) {
                const i32x2 cv = {(int)call, (int)(call - 1u)};
                if (lane == 0)
                    asm volatile("s_nop 4\n\tglobal_atomic_cmpswap %0, %1, %2, %3 sc0" : "+v"(seen) : "v"(0u), "v"(cv), "s"(slots + label * 32) : "memory");
            }
            readA(0, 4, sn);
            writeP();
            { const int t = sa; sa = sn; sn = t; }
            WAIT_L0();
            BAR();
        }
        BAR();
    } else {
#pragma unroll 1
        for (int i = 1; i < nblk; ++i) {
            phaseX(i, true);
            emitY();
            BAR();
            readY(0);
            if (__builtin_expect(i == 1, 0)) WAIT_V(9);
            WAIT_V(10);
            readA(0, 4, sn);
            writeP();
            { const int t = sa; sa = sn; sn = t; }
            po = (ps[0] + ps[1]) + (ps[2] + ps[3]);
            WAIT_L0();
            BAR();
        }
        readY(4); emitY();
    }

    WAIT_V(0);
    if (!owned && wv == 0) {
        asm volatile("" : "+v"(seen));
        *(volatile LAS int*)smem = __builtin_amdgcn_readfirstlane(seen) == call - 1u ? label : reclaim(call);
        WAIT_L0();
    }
    BAR();
    if (tid == 0 && dst != rank) __hip_atomic_store((unsigned*)(P.ws[lab] + WS_FLAG) + rank * 32 + ti, call, RLX_SYS);
    if (!owned) {
        const int l = __builtin_amdgcn_readfirstlane(*(volatile LAS int*)smem);
        owned = true;
        if (l != label) { WAIT_L0(); BAR(); label = l; continue; }
    }
    if (wv != 0) return;
    if (lane == 0) __hip_atomic_store(cnt, call, RLX_AGT);
    if (!remote && ti == 31) {
        GAS char* fl = myws + WS_FLAG;
        const bool skip = (lane >> 3) == rank || (lane & 7) >= 4;
        const unsigned long long t0 = __builtin_amdgcn_s_memrealtime();
        for (;;) {
            u32x4 f;
            asm volatile("s_nop 4\n\tglobal_load_dwordx4 %0, %1, %2 sc0 sc1\n\ts_waitcnt vmcnt(0)" : "=v"(f) : "v"(lane * 16), "s"(fl) : "memory");
            const bool ok = skip || (f.x >= call && f.y >= call && f.z >= call && f.w >= call);
            if (__all(ok)) break;
            if (__any(lane == rank * 8 && f.x == call) || __builtin_amdgcn_s_memrealtime() - t0 > DEADLINE_TICKS) {
                if (lane == 0) {
                    __hip_atomic_store(tmo, call, RLX_AGT);
                    __hip_atomic_store(P.myout + (size_t)rank * MLOC * NLOC, __builtin_nanf(""), RLX_SYS);
                }
                break;
            }
            __builtin_amdgcn_s_sleep(1);
        }
    }
    return;
  }
}

extern "C" void kernel_launch(int rank, int nranks, ncclComm_t comm,
                              hipStream_t stream,
                              void* const* d_in, const int* in_sizes, int n_in,
                              void* d_out, int out_size,
                              const int* in_dtypes, int out_dtype,
                              void* const* d_out_all,
                              void* const* d_ws_all, size_t ws_bytes) {
    if (nranks != NR || (unsigned)rank >= (unsigned)NR || n_in < 4 || ws_bytes < WS_NEED ||
        in_sizes[0] != MLOC * KDIM || in_sizes[1] != KDIM * NDIM || out_size != NR * MLOC * NLOC) {
        fprintf(stderr, "kernel_launch: unexpected problem shape\n");
        return;
    }
    static bool attr_set = false;
    if (!attr_set) {
        HIP_CHECK(hipFuncSetAttribute((const void*)gemm_a2a, hipFuncAttributeMaxDynamicSharedMemorySize, LDS_BYTES));
        attr_set = true;
    }
    Params P;
    P.x = (const uint8_t*)d_in[0];
    P.w = (const uint8_t*)d_in[1];
    P.sx = (const float*)d_in[2];
    P.sw = (const float*)d_in[3];
    for (int p = 0; p < NR; ++p) { P.out[p] = (float*)d_out_all[p]; P.ws[p] = (char*)d_ws_all[p]; }
    P.myws = P.ws[rank];
    P.myout = P.out[rank];
    P.rank = rank;
    {
        const int half[2][4] = {{0, 2, 3, 6}, {1, 4, 5, 7}};
        const int h = (rank == 1 || rank == 4 || rank == 5 || rank == 7);
        P.lmap = 0;
        for (int j = 0; j < 8; ++j) P.lmap |= (unsigned)half[j < 4 ? h : 1 - h][j & 3] << (4 * j);
    }
    hipLaunchKernelGGL(gemm_a2a, dim3(NWG), dim3(512), LDS_BYTES, stream, P);
    HIP_CHECK(hipGetLastError());
}
```
